```python
import numpy as np

kernel_name = "gpu_ag_gemm_m2048_k2048_n2048_f32_relu"


ABSMAX_RTOL = 0.001

COMPUTE_DTYPE = "float32"
EPILOGUE = "relu"
IN_QUANT = None
OUT_QUANT = None
IN_DTYPES = {"x": "float32", "w_mat": "float32"}
WORLD_SIZE = 8

def setup_inputs():

    x = np.random.randn(2048, 2048).astype(np.float32)
    w_mat = (np.random.randn(2048, 2048) / np.sqrt(2048)).astype(np.float32)
    return {"x": x, "w_mat": w_mat}

def _epilogue(y):
    return np.maximum(y, 0.0).astype(np.float32)

def reference(x, w_mat):
    y = (x @ w_mat).astype(np.float32)
    return _epilogue(y)
```

```cpp
#include <hip/hip_runtime.h>
#include <rccl/rccl.h>
#include <cstdio>
#include <cstdint>

#define HIP_CHECK(expr) do { hipError_t _e = (expr); if (_e != hipSuccess) { \
    fprintf(stderr, "HIP error %s at %s:%d\n", hipGetErrorString(_e), __FILE__, __LINE__); \
    return; } } while (0)

#define GAS __attribute__((address_space(1)))
#define LAS __attribute__((address_space(3)))
typedef _Float16 f16;
typedef f16 f16x8 __attribute__((ext_vector_type(8)));
typedef f16 f16x2 __attribute__((ext_vector_type(2)));
typedef float f32x4 __attribute__((ext_vector_type(4)));
typedef float f32x2 __attribute__((ext_vector_type(2)));
typedef unsigned u32x4 __attribute__((ext_vector_type(4)));
typedef GAS unsigned gu32;

constexpr int NR = 8;
constexpr int KD = 2048;
constexpr int NL = 256;
constexpr int ML = 256;
constexpr int NWG = 256;
constexpr int NPH = 4;
constexpr int NGW = 4;
constexpr int NSW = 2;
constexpr int NTHREADS = 64 * (NGW + NSW);

constexpr size_t WS_CNT  = 0;
constexpr size_t WS_TMO  = 32768;
constexpr size_t WS_FLAG = 65536;
constexpr size_t WS_SLOT = 1u << 20;
constexpr size_t SRC_BYTES  = (size_t)ML * KD * 2;
constexpr size_t SLOT_BYTES = SRC_BYTES * NR;
constexpr size_t WS_NEED = WS_SLOT + 2 * SLOT_BYTES;
constexpr unsigned long long DEADLINE_TICKS = 50000000ull;

constexpr int LDS_BYTES = 96 * 1024;

struct Params {
    const float* x;
    const float* w;
    float* out;
    char* ws[NR];
    int rank;
    int pad;
};

__device__ __forceinline__ void store16_sys(void* p, u32x4 v) {
    asm volatile("global_store_dwordx4 %0, %1, off sc0 sc1\n\ts_nop 1" :: "v"(p), "v"(v) : "memory");
}
__device__ __forceinline__ unsigned pack2(float a, float b) {
    f16x2 h = {(f16)a, (f16)b};
    return __builtin_bit_cast(unsigned, h);
}

__global__ void __launch_bounds__(NTHREADS, 2) ag_gemm(Params P) {
    extern __shared__ __attribute__((aligned(16))) char smem[];
    const int wg = blockIdx.x;
    const int wave = __builtin_amdgcn_readfirstlane(threadIdx.x >> 6);
    const int lane = threadIdx.x & 63;
    char* myws = P.ws[P.rank];

    LAS unsigned* lcnt = (LAS unsigned*)smem;
    if (threadIdx.x < 16) lcnt[threadIdx.x] = 0u;

    gu32* cntp = (gu32*)(myws + WS_CNT + (size_t)wg * 128);
    const unsigned call = __builtin_amdgcn_readfirstlane(__hip_atomic_load(cntp, __ATOMIC_RELAXED, __HIP_MEMORY_SCOPE_AGENT)) + 1u;
    __syncthreads();
    if (threadIdx.x == 0) __hip_atomic_store(cntp, call, __ATOMIC_RELAXED, __HIP_MEMORY_SCOPE_AGENT);
    const unsigned seq0 = (call - 1u) * NPH;
    const size_t slot_off = WS_SLOT + (size_t)(call & 1u) * SLOT_BYTES;

    if (wave >= NGW) {
        const int a = wave - NGW;
        const int d = (P.rank + (wg & 7)) & 7;
        const int u = wg >> 3, bb = u >> 3, c = u & 7;
        char* dws = P.ws[d];
        u32x4 fr[2][8];
#pragma unroll
        for (int pp = 0; pp < 2; ++pp) {
            const int blk = (a + 2 * pp) * 4 + bb;
            const float* src = P.x + (size_t)(blk * 16 + (lane & 15)) * KD + c * 256 + (lane >> 4) * 8;
#pragma unroll
            for (int s = 0; s < 8; ++s) {
                const f32x4 v0 = *(const f32x4*)(src + s * 32);
                const f32x4 v1 = *(const f32x4*)(src + s * 32 + 4);
                fr[pp][s] = (u32x4){pack2(v0[0], v0[1]), pack2(v0[2], v0[3]), pack2(v1[0], v1[1]), pack2(v1[2], v1[3])};
            }
        }
        gu32* flag = (gu32*)(dws + WS_FLAG + (size_t)(P.rank * 32 + u) * 128);
        flag += a * 16;
        if (a == 1) __builtin_amdgcn_s_sleep(20);
#pragma unroll
        for (int pp = 0; pp < 2; ++pp) {
            const int ph = a + 2 * pp;
            const int blk = ph * 4 + bb;
            char* dst = dws + slot_off + (size_t)P.rank * SRC_BYTES + (size_t)blk * 65536 + (size_t)c * 8192 + lane * 16;
#pragma unroll
            for (int s = 0; s < 8; ++s) store16_sys(dst + s * 1024, fr[pp][s]);
            asm volatile("s_waitcnt vmcnt(0)" ::: "memory");
            if (lane == 0) __hip_atomic_store(flag, seq0 + ph + 1u, __ATOMIC_RELAXED, __HIP_MEMORY_SCOPE_SYSTEM);
        }
        return;
    }

    const int q = wave;
    const int xcd = wg & 7, t = wg >> 3, g = t & 7, bb = t >> 3;
    const int s = (P.rank + xcd) & 7;
    f16x8 b0[16], b1[16];
    {
        const float* wp = P.w + (size_t)(q * 512 + (lane >> 4) * 8) * NL + g * 32 + (lane & 15) * 2;
#pragma unroll
        for (int ks = 0; ks < 16; ++ks) {
            f32x2 v[8];
#pragma unroll
            for (int j = 0; j < 8; ++j) v[j] = *(const f32x2*)(wp + (size_t)(ks * 32 + j) * NL);
#pragma unroll
            for (int j = 0; j < 8; ++j) { b0[ks][j] = (f16)v[j][0]; b1[ks][j] = (f16)v[j][1]; }
            asm volatile("" : "+v"(b0[ks]), "+v"(b1[ks]));
            if ((ks & 3) == 3) __builtin_amdgcn_sched_barrier(0);
        }
    }
    gu32* tmo = (gu32*)(myws + WS_TMO);
    const __amdgpu_buffer_rsrc_t rs = __builtin_amdgcn_make_buffer_rsrc(
        myws + slot_off + (size_t)s * SRC_BYTES, 0, (int)SRC_BYTES, 0x00020000);
    for (int ph = 0; ph < NPH; ++ph) {
        gu32* flag = (gu32*)(myws + WS_FLAG + (size_t)(s * 32 + bb * 8 + 2 * q + (lane & 1)) * 128) + (ph & 1) * 16;
        const unsigned need = seq0 + ph + 1u;
        bool ok = true;
        {
            const unsigned long long t0 = __builtin_amdgcn_s_memrealtime();
            for (;;) {
                const unsigned v = __hip_atomic_load(flag, __ATOMIC_RELAXED, __HIP_MEMORY_SCOPE_SYSTEM);
                if (__all((int)(v - need) >= 0)) break;
                if (__builtin_amdgcn_readfirstlane(__hip_atomic_load(tmo, __ATOMIC_RELAXED, __HIP_MEMORY_SCOPE_AGENT)) == call
                    || __builtin_amdgcn_s_memrealtime() - t0 > DEADLINE_TICKS) {
                    if (lane == 0) __hip_atomic_store(tmo, call, __ATOMIC_RELAXED, __HIP_MEMORY_SCOPE_AGENT);
                    ok = false;
                    break;
                }
                __builtin_amdgcn_s_sleep(2);
            }
        }
        asm volatile("" ::: "memory");
        const int blk = ph * 4 + bb;
        f32x4 acc0 = {0.f, 0.f, 0.f, 0.f}, acc1 = {0.f, 0.f, 0.f, 0.f};
        const unsigned tv = __hip_atomic_load(tmo, __ATOMIC_RELAXED, __HIP_MEMORY_SCOPE_AGENT);
        if (ok) {
            u32x4 av[16];
#pragma unroll
            for (int ks = 0; ks < 16; ++ks)
                av[ks] = __builtin_amdgcn_raw_buffer_load_b128(rs, blk * 65536 + (q * 16 + ks) * 1024 + lane * 16, 0, 17);
#pragma unroll
            for (int ks = 0; ks < 16; ++ks) {
                const f16x8 af = __builtin_bit_cast(f16x8, av[ks]);
                acc0 = __builtin_amdgcn_mfma_f32_16x16x32_f16(af, b0[ks], acc0, 0, 0, 0);
                acc1 = __builtin_amdgcn_mfma_f32_16x16x32_f16(af, b1[ks], acc1, 0, 0, 0);
            }
        }
        if (!ok || tv == call) {
            const float nanv = __builtin_nanf("");
            acc0 = (f32x4){nanv, nanv, nanv, nanv};
            acc1 = acc0;
        }
        LAS f32x4* part = (LAS f32x4*)(smem + 1024 + ph * 8192);
        part[(q * 2 + 0) * 64 + lane] = acc0;
        part[(q * 2 + 1) * 64 + lane] = acc1;
        asm volatile("s_waitcnt lgkmcnt(0)" ::: "memory");
        unsigned old = 0;
        if (lane == 0) old = __hip_atomic_fetch_add(&lcnt[ph], 1u, __ATOMIC_ACQ_REL, __HIP_MEMORY_SCOPE_WORKGROUP);
        old = __builtin_amdgcn_readfirstlane(old);
        if (old == NGW - 1) {
            f32x4 r0 = part[0 * 64 + lane] + part[2 * 64 + lane] + part[4 * 64 + lane] + part[6 * 64 + lane];
            f32x4 r1 = part[1 * 64 + lane] + part[3 * 64 + lane] + part[5 * 64 + lane] + part[7 * 64 + lane];
            float* op = P.out + (size_t)(s * ML + blk * 16 + (lane >> 4) * 4) * NL + g * 32 + (lane & 15) * 2;
#pragma unroll
            for (int r = 0; r < 4; ++r) {
                const float y0 = r0[r], y1 = r1[r];
                f32x2 o = {y0 < 0.f ? 0.f : y0, y1 < 0.f ? 0.f : y1};
                *(f32x2*)(op + (size_t)r * NL) = o;
            }
        }
    }
}

extern "C" void kernel_launch(int rank, int nranks, ncclComm_t comm,
                              hipStream_t stream,
                              void* const* d_in, const int* in_sizes, int n_in,
                              void* d_out, int out_size,
                              const int* in_dtypes, int out_dtype,
                              void* const* d_out_all,
                              void* const* d_ws_all, size_t ws_bytes) {
    (void)comm; (void)in_dtypes; (void)out_dtype; (void)d_out_all;
    if (nranks != NR || n_in != 2 || in_sizes[0] != ML * KD || in_sizes[1] != KD * NL ||
        out_size != NR * ML * NL || ws_bytes < WS_NEED || (unsigned)rank >= (unsigned)NR) {
        fprintf(stderr, "kernel_launch: unexpected shapes\n");
        return;
    }
    static bool attr_set[64] = {};
    int dev = 0;
    HIP_CHECK(hipGetDevice(&dev));
    if (dev >= 0 && dev < 64 && !attr_set[dev]) {
        HIP_CHECK(hipFuncSetAttribute((const void*)ag_gemm, hipFuncAttributeMaxDynamicSharedMemorySize, LDS_BYTES));
        attr_set[dev] = true;
    }
    Params p;
    p.x = (const float*)d_in[0];
    p.w = (const float*)d_in[1];
    p.out = (float*)d_out;
    for (int i = 0; i < NR; ++i) p.ws[i] = (char*)d_ws_all[i];
    p.rank = rank;
    p.pad = 0;
    hipLaunchKernelGGL(ag_gemm, dim3(NWG), dim3(NTHREADS), LDS_BYTES, stream, p);
    HIP_CHECK(hipGetLastError());
}
```

```cpp
#include <hip/hip_runtime.h>
#include <rccl/rccl.h>
#include <cstdio>
#include <cstdint>
#include <type_traits>

#define HIP_CHECK(expr) do { hipError_t _e = (expr); if (_e != hipSuccess) { \
    fprintf(stderr, "HIP error %s at %s:%d\n", hipGetErrorString(_e), __FILE__, __LINE__); \
    return; } } while (0)

#define GAS __attribute__((address_space(1)))
#define LAS __attribute__((address_space(3)))
typedef _Float16 f16;
typedef f16 f16x8 __attribute__((ext_vector_type(8)));
typedef f16 f16x2 __attribute__((ext_vector_type(2)));
typedef float f32x4 __attribute__((ext_vector_type(4)));
typedef float f32x2 __attribute__((ext_vector_type(2)));
typedef unsigned u32x4 __attribute__((ext_vector_type(4)));
typedef unsigned short u16x2 __attribute__((ext_vector_type(2)));
typedef unsigned short u16x8 __attribute__((ext_vector_type(8)));
typedef GAS unsigned gu32;
typedef __amdgpu_buffer_rsrc_t rsrc_t;

constexpr int NR = 8;
constexpr int KD = 2048;
constexpr int NL = 256;
constexpr int ML = 256;
constexpr int NWG = 256;
constexpr int NWAVE = 6;
constexpr int NTHREADS = 64 * NWAVE;
#define NI_BLK 11
constexpr int NI = NI_BLK;
constexpr int NX = 16 - NI;
constexpr int NSEND = 32;

constexpr unsigned XBLK = 65536;
constexpr unsigned WGRP = 131072;
constexpr unsigned IN_OFF = 0;
constexpr unsigned CR_OFF = 4 * NI * XBLK;
constexpr unsigned CR_SZ = 2 * WGRP + NX * XBLK;
constexpr unsigned OW_OFF = CR_OFF + 4 * CR_SZ;
constexpr unsigned SLOT_BYTES = OW_OFF + 8 * WGRP;
static_assert(SLOT_BYTES == 6u << 20, "slot size");
constexpr size_t WS_CNT  = 0;
constexpr size_t WS_TMO  = 32768;
constexpr size_t WS_FLAG = 65536;
constexpr size_t WS_SLOT = 1u << 20;
constexpr size_t WS_NEED = WS_SLOT + 3 * (size_t)SLOT_BYTES;
constexpr unsigned long long DEADLINE_TICKS = 40000000ull;
constexpr unsigned long long SLOW_TICKS = 20000ull;

constexpr int LDS_BYTES = 96 * 1024;
constexpr int LDS_PART = 1024;
constexpr int LDS_TR = 1024;
constexpr int LDS_TR5 = LDS_PART + 6 * 8192;
#define ZDELAY 3
#define ZPACE 10
#define WIN_IN 3
#define WIN_X 0

struct Order8 { unsigned char e[64]; };
struct Order { unsigned w[16]; };
constexpr Order make_order() {
    Order8 o{};
    int t[64] = {};
    int n = 0;
    for (int b = 0; b < NI; ++b) { o.e[n] = (unsigned char)b; t[n] = 0; ++n; }
    for (int b = 0; b < NI; ++b) for (int j = 1; j < 4; ++j) { o.e[n] = (unsigned char)(j << 4 | b); t[n] = 10 * (b + 1); ++n; }
    for (int b = NI; b < 16; ++b) for (int c = 0; c < 4; ++c) {
        const int r = (b - NI) >> 1;
        o.e[n] = (unsigned char)(1 << 6 | c << 4 | b);
        t[n] = (r == 2) ? 106 : 41 + 25 * (r + 1);
        ++n;
    }
    for (int i = 1; i < 64; ++i) {
        const unsigned char e = o.e[i]; const int ti = t[i];
        int j = i - 1;
        while (j >= 0 && t[j] > ti) { o.e[j + 1] = o.e[j]; t[j + 1] = t[j]; --j; }
        o.e[j + 1] = e; t[j + 1] = ti;
    }
    Order r{};
    for (int i = 0; i < 64; ++i) r.w[i >> 2] |= (unsigned)o.e[i] << (8 * (i & 3));
    return r;
}
__constant__ const Order ORDER = make_order();

struct Params {
    const float* x;
    const float* w;
    char* ws[NR];
    char* out[NR];
    int rank;
    int pad;
};

template <int I, int N, class F> __device__ __forceinline__ void static_for(F&& f) {
    if constexpr (I < N) { f(std::integral_constant<int, I>{}); static_for<I + 1, N>(f); }
}
template <int N> __device__ __forceinline__ void wait_vm() { asm volatile("s_waitcnt vmcnt(%0)" :: "n"(N) : "memory"); }
__device__ __forceinline__ rsrc_t mk_rsrc(const void* p, unsigned bytes) {
    return __builtin_amdgcn_make_buffer_rsrc((void*)p, 0, (int)bytes, 0x00020000);
}
constexpr int AUX_SYS = 17;
__device__ __forceinline__ void st16(u32x4 v, rsrc_t r, unsigned voff, unsigned soff) {
    __builtin_amdgcn_raw_buffer_store_b128(v, r, (int)(voff + soff), 0, AUX_SYS);
}
__device__ __forceinline__ u32x4 ld16(rsrc_t r, unsigned voff, unsigned soff) {
    return __builtin_amdgcn_raw_buffer_load_b128(r, (int)voff, (int)soff, AUX_SYS);
}
__device__ __forceinline__ unsigned ld4(rsrc_t r, unsigned voff, unsigned soff) {
    return __builtin_amdgcn_raw_buffer_load_b32(r, (int)voff, (int)soff, AUX_SYS);
}
__device__ __forceinline__ unsigned pack2(float a, float b) {
    const f16x2 h = {(f16)a, (f16)b};
    const u16x2 u = __builtin_bit_cast(u16x2, h);
    const u16x2 one = {1, 1};
    const u16x2 z = __builtin_elementwise_sub_sat(one, u);
    const u16x2 r = u | (u16x2)(z << 15);
    return __builtin_bit_cast(unsigned, r);
}
__device__ __forceinline__ bool all_nonzero(u16x8 m) {
    const u16x2 a = __builtin_elementwise_min((u16x2){m[0], m[1]}, (u16x2){m[2], m[3]});
    const u16x2 b = __builtin_elementwise_min((u16x2){m[4], m[5]}, (u16x2){m[6], m[7]});
    const u16x2 c = __builtin_elementwise_min(a, b);
    return __all(c[0] != 0 && c[1] != 0);
}

#define STAMP_DECL
#define STAMP(i)
#define STAMP_FLUSH()

__device__ __forceinline__ bool give_up(unsigned long long t0, gu32* tmo, unsigned call, int lane) {
    const unsigned long long dt = __builtin_amdgcn_s_memrealtime() - t0;
    if (dt > SLOW_TICKS) {
        const unsigned tv = __builtin_amdgcn_readfirstlane(__hip_atomic_load(tmo, __ATOMIC_RELAXED, __HIP_MEMORY_SCOPE_AGENT));
        if (tv == call || dt > DEADLINE_TICKS) {
            if (lane == 0) __hip_atomic_store(tmo, call, __ATOMIC_RELAXED, __HIP_MEMORY_SCOPE_AGENT);
            return true;
        }
    }
    return false;
}

__device__ __forceinline__ u32x4 x_fragment(const f32x4 lo, const f32x4 hi, LAS char* tr, int lane) {
    const u32x4 v = {pack2(lo[0], lo[1]), pack2(lo[2], lo[3]), pack2(hi[0], hi[1]), pack2(hi[2], hi[3])};
    *(LAS u32x4*)(tr + (lane >> 2) * 16 + (lane & 3) * 320) = v;
    return *(LAS u32x4*)(tr + (lane & 15) * 16 + (lane >> 4) * 320);
}

__global__ void __launch_bounds__(NTHREADS, 2) ag_gemm(Params P) {
    extern __shared__ __attribute__((aligned(16))) char smem_[];
    LAS char* smem = (LAS char*)smem_;
    const int wg = blockIdx.x;
    const int wave = __builtin_amdgcn_readfirstlane(threadIdx.x >> 6);
    const int lane = threadIdx.x & 63;
    const int p = P.rank, pa = p & 3, cls = p & 4;
    char* myws = P.ws[p];
    STAMP_DECL;
    STAMP(0);
    gu32* cntp = (gu32*)(myws + WS_CNT + (size_t)wg * 128 + wave * 4);
    gu32* tmo = (gu32*)(myws + WS_TMO);

    if (wg < NSEND) {
        LAS char* tr = smem + LDS_TR + wave * 1280;
        if (wave < 2) {
            const int ks = wg * 2 + wave;
            const unsigned cv = __hip_atomic_load(cntp, __ATOMIC_RELAXED, __HIP_MEMORY_SCOPE_AGENT);
            const float* xp = P.x + (size_t)(lane >> 2) * KD + ks * 32 + (lane & 3) * 4;
            f32x4 raw[NI][2];
#pragma unroll
            for (int b = 0; b < NI; ++b) {
                raw[b][0] = *(const f32x4*)(xp + (size_t)b * 16 * KD);
                raw[b][1] = *(const f32x4*)(xp + (size_t)b * 16 * KD + 16);
            }
            __builtin_amdgcn_sched_barrier(0);
            __builtin_amdgcn_s_barrier();
            __builtin_amdgcn_sched_barrier(0);
            const unsigned call = __builtin_amdgcn_readfirstlane(cv) + 1u;
            if (lane == 0) __hip_atomic_store(cntp, call, __ATOMIC_RELAXED, __HIP_MEMORY_SCOPE_AGENT);
            const size_t cur = WS_SLOT + (size_t)(call % 3u) * SLOT_BYTES;
            const rsrc_t r1 = mk_rsrc(P.ws[cls | ((p + 1) & 3)] + cur, SLOT_BYTES);
            const rsrc_t r2 = mk_rsrc(P.ws[cls | ((p + 2) & 3)] + cur, SLOT_BYTES);
            const rsrc_t r3 = mk_rsrc(P.ws[cls | ((p + 3) & 3)] + cur, SLOT_BYTES);
            const unsigned voff = ks * 1024 + lane * 16;
            STAMP(1);
            static_for<0, NI>([&](auto B) {
                constexpr int b = B;
                const u32x4 fr = x_fragment(raw[b][0], raw[b][1], tr, lane);
                if constexpr (b >= 2) wait_vm<WIN_IN>();
                const unsigned so = IN_OFF + (pa * NI + b) * XBLK;
                st16(fr, r1, voff, so);
                st16(fr, r2, voff, so);
                st16(fr, r3, voff, so);
                __builtin_amdgcn_sched_barrier(0);
                if constexpr (b == 0) STAMP(2);
                if constexpr (b == 2) STAMP(3);
                if constexpr (b == 5) STAMP(4);
                if constexpr (b == NI - 1) STAMP(5);
            });
            if (wg == 0 && wave == 0) {
                const rsrc_t rf = mk_rsrc(myws + WS_FLAG, 128 * 128);
                const unsigned long long t0 = __builtin_amdgcn_s_memrealtime();
                for (;;) {
                    const unsigned v0 = ld4(rf, lane * 128, 0), v1 = ld4(rf, lane * 128 + 8192, 0);
                    asm volatile("" ::: "memory");
                    if (__all((int)(v0 - call) >= 0 && (int)(v1 - call) >= 0)) break;
                    if (give_up(t0, tmo, call, lane)) {
                        if (lane == 0) *(GAS float*)(P.out[p] + (size_t)p * ML * NL * 4) = __builtin_nanf("");
                        break;
                    }
                    __builtin_amdgcn_s_sleep(1);
                }
                STAMP(7);
            }
            STAMP_FLUSH();
            return;
        }
        const int v = wg * 4 + (wave - 2), c = v & 3, i = v >> 2;
        const unsigned cv = __hip_atomic_load(cntp, __ATOMIC_RELAXED, __HIP_MEMORY_SCOPE_AGENT);
        f32x2 wr[4][8];
#pragma unroll
        for (int pr = 0; pr < 4; ++pr) {
            const int g = 2 * c + (pr >> 1), ks = i + 32 * (pr & 1);
            const float* wp = P.w + (size_t)(ks * 32 + (lane >> 4) * 4) * NL + g * 32 + (lane & 15) * 2;
#pragma unroll
            for (int j = 0; j < 8; ++j) wr[pr][j] = *(const f32x2*)(wp + (size_t)((j & 3) + (j >> 2) * 16) * NL);
        }
        constexpr int NXI = (NX * 64 + 127) / 128;
        f32x4 xr[NXI][2];
#pragma unroll
        for (int it = 0; it < NXI; ++it) {
            int item = v + 128 * it;
            if (item >= NX * 64) item = v;
            const float* xp = P.x + (size_t)((NI + (item >> 6)) * 16 + (lane >> 2)) * KD + (item & 63) * 32 + (lane & 3) * 4;
            xr[it][0] = *(const f32x4*)xp;
            xr[it][1] = *(const f32x4*)(xp + 16);
        }
        __builtin_amdgcn_sched_barrier(0);
        __builtin_amdgcn_s_barrier();
        __builtin_amdgcn_sched_barrier(0);
        const unsigned call = __builtin_amdgcn_readfirstlane(cv) + 1u;
        if (lane == 0) __hip_atomic_store(cntp, call, __ATOMIC_RELAXED, __HIP_MEMORY_SCOPE_AGENT);
        const size_t cur = WS_SLOT + (size_t)(call % 3u) * SLOT_BYTES;
        const int oc = cls ^ 4;
        const rsrc_t rc = mk_rsrc(P.ws[oc | c] + cur, SLOT_BYTES);
        u32x4 wf[4][2];
        auto cvt_pair = [&](int pr) {
            wf[pr][0] = (u32x4){pack2(wr[pr][0][0], wr[pr][1][0]), pack2(wr[pr][2][0], wr[pr][3][0]), pack2(wr[pr][4][0], wr[pr][5][0]), pack2(wr[pr][6][0], wr[pr][7][0])};
            wf[pr][1] = (u32x4){pack2(wr[pr][0][1], wr[pr][1][1]), pack2(wr[pr][2][1], wr[pr][3][1]), pack2(wr[pr][4][1], wr[pr][5][1]), pack2(wr[pr][6][1], wr[pr][7][1])};
        };
        const unsigned l16 = lane * 16;
        auto send_pair = [&](int pr) {
            const unsigned so = CR_OFF + pa * CR_SZ + (pr >> 1) * WGRP + (i + 32 * (pr & 1)) * 2048;
            st16(wf[pr][0], rc, l16, so);
            st16(wf[pr][1], rc, l16, so + 1024);
        };
        STAMP(1);
        cvt_pair(0); send_pair(0);
        __builtin_amdgcn_sched_barrier(0);
        STAMP(2);
        cvt_pair(1); send_pair(1);
        __builtin_amdgcn_sched_barrier(0);
        cvt_pair(2); cvt_pair(3);
        u32x4 xf[NXI];
#pragma unroll
        for (int it = 0; it < NXI; ++it) xf[it] = x_fragment(xr[it][0], xr[it][1], tr, lane);
        __builtin_amdgcn_sched_barrier(0);
        wait_vm<2>(); send_pair(2);
        __builtin_amdgcn_sched_barrier(0);
        wait_vm<2>(); send_pair(3);
        __builtin_amdgcn_sched_barrier(0);
        STAMP(3);
        const rsrc_t q0 = mk_rsrc(P.ws[oc | 0] + cur, SLOT_BYTES), q1 = mk_rsrc(P.ws[oc | 1] + cur, SLOT_BYTES);
        const rsrc_t q2 = mk_rsrc(P.ws[oc | 2] + cur, SLOT_BYTES), q3 = mk_rsrc(P.ws[oc | 3] + cur, SLOT_BYTES);
        static_for<0, NXI>([&](auto IT) {
            constexpr int it = IT;
            const int item = v + 128 * it;
            if (item < NX * 64) {
                wait_vm<WIN_X>();
                const unsigned so = CR_OFF + pa * CR_SZ + 2 * WGRP + (item >> 6) * XBLK + (item & 63) * 1024;
                st16(xf[it], q0, l16, so);
                st16(xf[it], q1, l16, so);
                st16(xf[it], q2, l16, so);
                st16(xf[it], q3, l16, so);
            }
            __builtin_amdgcn_sched_barrier(0);
            if constexpr (it == 0) STAMP(4);
            if constexpr (it == 1) STAMP(5);
        });
        STAMP(6);
        STAMP_FLUSH();
        return;
    }

    const int xcd = wg & 7, u = (wg >> 3) - NSEND / 8, rq = xcd >> 1;
    const bool remote = u < 16;
    const int uu = remote ? u : u - 16;
    const int cg = 4 * (xcd & 1) + (uu & 3), m = uu >> 2;
    const int nblk = remote ? 4 : (m == 0 ? 6 : 5), jstep = remote ? 4 : 3;
    LAS unsigned* lcnt = (LAS unsigned*)smem;
    const unsigned cv = __hip_atomic_load(cntp, __ATOMIC_RELAXED, __HIP_MEMORY_SCOPE_AGENT);
    if (wave == 5) {
        LAS char* tr = smem + LDS_TR5;
        if (lane < 8) lcnt[lane] = 0u;
        if (remote) {
            const int g = u >> 1, ks0 = xcd + 32 * (u & 1);
            f32x2 wr[4][8];
#pragma unroll
            for (int pr = 0; pr < 4; ++pr) {
                const float* wp = P.w + (size_t)((ks0 + 8 * pr) * 32 + (lane >> 4) * 4) * NL + g * 32 + (lane & 15) * 2;
#pragma unroll
                for (int j = 0; j < 8; ++j) wr[pr][j] = *(const f32x2*)(wp + (size_t)((j & 3) + (j >> 2) * 16) * NL);
            }
            __builtin_amdgcn_sched_barrier(0);
            asm volatile("s_waitcnt lgkmcnt(0)" ::: "memory");
            __builtin_amdgcn_s_barrier();
            __builtin_amdgcn_sched_barrier(0);
            const unsigned call = __builtin_amdgcn_readfirstlane(cv) + 1u;
            if (lane == 0) __hip_atomic_store(cntp, call, __ATOMIC_RELAXED, __HIP_MEMORY_SCOPE_AGENT);
            const rsrc_t rl = mk_rsrc(myws + WS_SLOT + (size_t)(call % 3u) * SLOT_BYTES, SLOT_BYTES);
#pragma unroll
            for (int pr = 0; pr < 4; ++pr) {
                const u32x4 f0 = {pack2(wr[pr][0][0], wr[pr][1][0]), pack2(wr[pr][2][0], wr[pr][3][0]), pack2(wr[pr][4][0], wr[pr][5][0]), pack2(wr[pr][6][0], wr[pr][7][0])};
                const u32x4 f1 = {pack2(wr[pr][0][1], wr[pr][1][1]), pack2(wr[pr][2][1], wr[pr][3][1]), pack2(wr[pr][4][1], wr[pr][5][1]), pack2(wr[pr][6][1], wr[pr][7][1])};
                const unsigned so = OW_OFF + g * WGRP + (ks0 + 8 * pr) * 2048;
                st16(f0, rl, lane * 16, so);
                st16(f1, rl, lane * 16, so + 1024);
            }
        } else {
            static_assert(NI <= 12, "one block per own-column workgroup of an XCD");
            const bool has = uu < NI;
            const int b = has ? uu : 0;
            const float* xp = P.x + (size_t)(b * 16 + (lane >> 2)) * KD + xcd * 64 + (lane & 3) * 4;
            f32x4 xr[8][2];
#pragma unroll
            for (int it = 0; it < 8; ++it) {
                xr[it][0] = *(const f32x4*)(xp + ((it & 1) + (it >> 1) * 16) * 32);
                xr[it][1] = *(const f32x4*)(xp + ((it & 1) + (it >> 1) * 16) * 32 + 16);
            }
            __builtin_amdgcn_sched_barrier(0);
            asm volatile("s_waitcnt lgkmcnt(0)" ::: "memory");
            __builtin_amdgcn_s_barrier();
            __builtin_amdgcn_sched_barrier(0);
            const unsigned call = __builtin_amdgcn_readfirstlane(cv) + 1u;
            if (lane == 0) __hip_atomic_store(cntp, call, __ATOMIC_RELAXED, __HIP_MEMORY_SCOPE_AGENT);
            const rsrc_t rl = mk_rsrc(myws + WS_SLOT + (size_t)(call % 3u) * SLOT_BYTES, SLOT_BYTES);
            if (has) {
#pragma unroll
                for (int it = 0; it < 8; ++it) {
                    const u32x4 fr = x_fragment(xr[it][0], xr[it][1], tr, lane);
                    st16(fr, rl, lane * 16, IN_OFF + (pa * NI + b) * XBLK + (2 * xcd + (it & 1) + (it >> 1) * 16) * 1024);
                }
            }
        }
        STAMP(1);
        {
            const unsigned call = __builtin_amdgcn_readfirstlane(cv) + 1u;
            const rsrc_t rz = mk_rsrc(myws + WS_SLOT + (size_t)((call + 2u) % 3u) * SLOT_BYTES, SLOT_BYTES);
            const u32x4 zero = {0u, 0u, 0u, 0u};
            constexpr int T = SLOT_BYTES / 1024, NG = NWG - NSEND;
            for (int i = 0; i < ZDELAY; ++i) __builtin_amdgcn_s_sleep(16);
            for (unsigned f = wg - NSEND; f < (unsigned)T; f += NG) {
                st16(zero, rz, lane * 16, f * 1024);
                __builtin_amdgcn_s_sleep(ZPACE);
            }
        }
        STAMP(2);
        STAMP_FLUSH();
        return;
    }
    f32x4 pf[4];
    if (wave == 4) {
        const int li = (u < 24 ? u : 0) * 2;
#pragma unroll
        for (int k = 0; k < 2; ++k) {
            const int n = li + k, v = (xcd + 8 * (n / 12)) * 4 + (n % 12) / 3;
            int item = v + 128 * (n % 3);
            if (item >= NX * 64) item = v;
            const float* xp = P.x + (size_t)((NI + (item >> 6)) * 16 + (lane >> 2)) * KD + (item & 63) * 32 + (lane & 3) * 4;
            pf[2 * k] = *(const f32x4*)xp;
            pf[2 * k + 1] = *(const f32x4*)(xp + 16);
        }
    }
    asm volatile("s_waitcnt lgkmcnt(0)" ::: "memory");
    __builtin_amdgcn_s_barrier();
    const unsigned call = __builtin_amdgcn_readfirstlane(cv) + 1u;
    if (lane == 0) __hip_atomic_store(cntp, call, __ATOMIC_RELAXED, __HIP_MEMORY_SCOPE_AGENT);
    const size_t cur = WS_SLOT + (size_t)(call % 3u) * SLOT_BYTES;
    unsigned long long desc = 0;
#pragma unroll
    for (int t = 0; t < 6; ++t) {
        const int wi = m + jstep * t;
        desc |= (unsigned long long)((ORDER.w[wi < 15 ? wi : 15] >> (8 * rq)) & 255u) << (8 * t);
    }
    auto block_of = [&](int t, unsigned& xoff, unsigned& row0) {
        const unsigned e = (unsigned)(desc >> (8 * t)) & 255u;
        const unsigned src = (e >> 4) & 3u, b = e & 15u;
        if (e >> 6) { xoff = CR_OFF + src * CR_SZ + 2 * WGRP + (b - NI) * XBLK; row0 = ((cls ^ 4) | src) * ML + b * 16; }
        else { const unsigned a = (p + src) & 3u; xoff = IN_OFF + (a * NI + b) * XBLK; row0 = (cls | a) * ML + b * 16; }
    };

    if (wave == 4) {
        asm volatile("" :: "v"(pf[0]), "v"(pf[1]), "v"(pf[2]), "v"(pf[3]));
        const int c = cg >> 1, h = cg & 1;
        const int z = (rq * 2 + h) * 4 + m;
        const int dst = remote ? ((cls ^ 4) | c) : p;
        const unsigned col0 = remote ? 64 * pa + 32 * h : 32 * cg;
        const rsrc_t ro = mk_rsrc(P.out[dst], 2048 * NL * 4);
        const unsigned ovoff = (lane >> 3) * (NL * 4) + (col0 + (lane & 7) * 4) * 4;
        STAMP(1);
        for (int t = 0; t < nblk; ++t) {
            unsigned xoff, row0;
            block_of(t, xoff, row0);
            const unsigned long long t0 = __builtin_amdgcn_s_memrealtime();
            bool bad = false;
            while (__hip_atomic_load(&lcnt[t], __ATOMIC_ACQUIRE, __HIP_MEMORY_SCOPE_WORKGROUP) < 4u) {
                if (__builtin_amdgcn_s_memrealtime() - t0 > 4 * DEADLINE_TICKS) { bad = true; break; }
                __builtin_amdgcn_s_sleep(1);
            }
            LAS char* part = smem + LDS_PART + t * 8192 + lane * 16;
#pragma unroll
            for (int hh = 0; hh < 2; ++hh) {
                f32x4 s = *(LAS f32x4*)(part + hh * 1024) + *(LAS f32x4*)(part + hh * 1024 + 2048)
                        + *(LAS f32x4*)(part + hh * 1024 + 4096) + *(LAS f32x4*)(part + hh * 1024 + 6144);
#pragma unroll
                for (int r = 0; r < 4; ++r) {
                    s[r] = s[r] < 0.f ? 0.f : s[r];
                    if (bad) s[r] = __builtin_nanf("");
                }
                st16(__builtin_bit_cast(u32x4, s), ro, ovoff, (row0 + hh * 8) * (NL * 4));
            }
        }
        if (remote) {
            wait_vm<0>();
            if (lane == 0) __hip_atomic_store((gu32*)(P.ws[dst] + WS_FLAG + (size_t)(pa * 32 + z) * 128), call, __ATOMIC_RELAXED, __HIP_MEMORY_SCOPE_SYSTEM);
            STAMP(8);
        }
        STAMP_FLUSH();
        return;
    }

    const int q = wave;
    const unsigned woff = remote ? CR_OFF + (cg >> 1) * CR_SZ + (cg & 1) * WGRP : OW_OFF + cg * WGRP;
    const rsrc_t rs = mk_rsrc(myws + cur, SLOT_BYTES);
    const unsigned l16 = lane * 16;
    bool bad = false;
    f16x8 b0[16], b1[16];
    unsigned xoff, row0;
    block_of(0, xoff, row0);
    const unsigned xpoll = (q * 16 + (lane & 15)) * 1024 + 1020;
    unsigned pv;
    {
        const unsigned long long t0 = __builtin_amdgcn_s_memrealtime();
        const unsigned wpoll = (q * 32 + (lane & 31)) * 1024 + 1020;
        for (;;) {
            const unsigned v = ld4(rs, wpoll, woff);
            asm volatile("" ::: "memory");
            if (__all(v != 0u)) break;
            if (give_up(t0, tmo, call, lane)) { bad = true; break; }
            __builtin_amdgcn_s_sleep(2);
        }
        STAMP(1);
        for (;;) {
            u16x8 mn = {0xffff, 0xffff, 0xffff, 0xffff, 0xffff, 0xffff, 0xffff, 0xffff};
            pv = ld4(rs, xpoll, xoff);
#pragma unroll
            for (int ks = 0; ks < 16; ++ks) {
                const u32x4 w0 = ld16(rs, l16, woff + (q * 32 + ks * 2) * 1024);
                const u32x4 w1 = ld16(rs, l16, woff + (q * 32 + ks * 2 + 1) * 1024);
                b0[ks] = __builtin_bit_cast(f16x8, w0);
                b1[ks] = __builtin_bit_cast(f16x8, w1);
            }
#pragma unroll
            for (int ks = 0; ks < 16; ++ks) {
                mn = __builtin_elementwise_min(mn, __builtin_bit_cast(u16x8, b0[ks]));
                mn = __builtin_elementwise_min(mn, __builtin_bit_cast(u16x8, b1[ks]));
            }
            asm volatile("" ::: "memory");
            if (bad || all_nonzero(mn)) break;
            if (give_up(t0, tmo, call, lane)) { bad = true; break; }
        }
    }
    STAMP(2);
    for (int t = 0; t < nblk; ++t) {
        const unsigned long long t0 = __builtin_amdgcn_s_memrealtime();
        while (!bad && !__all(pv != 0u)) {
            if (give_up(t0, tmo, call, lane)) { bad = true; break; }
            __builtin_amdgcn_s_sleep(1);
            pv = ld4(rs, xpoll, xoff);
            asm volatile("" ::: "memory");
        }
        unsigned nxoff = xoff, nrow0;
        if (t + 1 < nblk) block_of(t + 1, nxoff, nrow0);
        f32x4 acc0, acc1;
        for (;;) {
            u32x4 av[16];
#pragma unroll
            for (int ks = 0; ks < 16; ++ks) av[ks] = ld16(rs, l16, xoff + (q * 16 + ks) * 1024);
            pv = ld4(rs, xpoll, nxoff);
            acc0 = (f32x4){0.f, 0.f, 0.f, 0.f};
            acc1 = acc0;
            u16x8 mn = {0xffff, 0xffff, 0xffff, 0xffff, 0xffff, 0xffff, 0xffff, 0xffff};
#pragma unroll
            for (int ks = 0; ks < 16; ++ks) {
                const f16x8 af = __builtin_bit_cast(f16x8, av[ks]);
                acc0 = __builtin_amdgcn_mfma_f32_16x16x32_f16(af, b0[ks], acc0, 0, 0, 0);
                acc1 = __builtin_amdgcn_mfma_f32_16x16x32_f16(af, b1[ks], acc1, 0, 0, 0);
                mn = __builtin_elementwise_min(mn, __builtin_bit_cast(u16x8, av[ks]));
            }
            asm volatile("" ::: "memory");
            if (bad || all_nonzero(mn)) break;
            if (give_up(t0, tmo, call, lane)) { bad = true; break; }
        }
        if (bad) {
            const float nanv = __builtin_nanf("");
            acc0 = (f32x4){nanv, nanv, nanv, nanv};
            acc1 = acc0;
        }
        LAS char* part = smem + LDS_PART + t * 8192 + q * 2048 + (lane >> 4) * 512 + (lane & 15) * 8;
#pragma unroll
        for (int r = 0; r < 4; ++r) *(LAS f32x2*)(part + r * 128) = (f32x2){acc0[r], acc1[r]};
        asm volatile("s_waitcnt lgkmcnt(0)" ::: "memory");
        if (lane == 0) __hip_atomic_fetch_add(&lcnt[t], 1u, __ATOMIC_RELEASE, __HIP_MEMORY_SCOPE_WORKGROUP);
        xoff = nxoff;
    }
    STAMP_FLUSH();
}

extern "C" void kernel_launch(int rank, int nranks, ncclComm_t comm,
                              hipStream_t stream,
                              void* const* d_in, const int* in_sizes, int n_in,
                              void* d_out, int out_size,
                              const int* in_dtypes, int out_dtype,
                              void* const* d_out_all,
                              void* const* d_ws_all, size_t ws_bytes) {
    (void)comm; (void)in_dtypes; (void)out_dtype; (void)d_out;
    if (nranks != NR || n_in != 2 || in_sizes[0] != ML * KD || in_sizes[1] != KD * NL ||
        out_size != NR * ML * NL || ws_bytes < WS_NEED || (unsigned)rank >= (unsigned)NR) {
        fprintf(stderr, "kernel_launch: unexpected shapes\n");
        return;
    }
    static bool attr_set[64] = {};
    int dev = 0;
    HIP_CHECK(hipGetDevice(&dev));
    if (dev >= 0 && dev < 64 && !attr_set[dev]) {
        HIP_CHECK(hipFuncSetAttribute((const void*)ag_gemm, hipFuncAttributeMaxDynamicSharedMemorySize, LDS_BYTES));
        attr_set[dev] = true;
    }
    Params p;
    p.x = (const float*)d_in[0];
    p.w = (const float*)d_in[1];
    for (int i = 0; i < NR; ++i) { p.ws[i] = (char*)d_ws_all[i]; p.out[i] = (char*)d_out_all[i]; }
    p.rank = rank;
    p.pad = 0;
    hipLaunchKernelGGL(ag_gemm, dim3(NWG), dim3(NTHREADS), LDS_BYTES, stream, p);
    HIP_CHECK(hipGetLastError());
}
```

```cpp
#include <hip/hip_runtime.h>
#include <rccl/rccl.h>
#include <cstdio>
#include <cstdint>
#include <cstddef>
#include <type_traits>

#define HIP_CHECK(expr) do { hipError_t _e = (expr); if (_e != hipSuccess) { \
    fprintf(stderr, "HIP error %s at %s:%d\n", hipGetErrorString(_e), __FILE__, __LINE__); \
    return; } } while (0)

#define GAS __attribute__((address_space(1)))
#define LAS __attribute__((address_space(3)))
typedef _Float16 f16;
typedef f16 f16x8 __attribute__((ext_vector_type(8)));
typedef f16 f16x2 __attribute__((ext_vector_type(2)));
typedef float f32x4 __attribute__((ext_vector_type(4)));
typedef float f32x2 __attribute__((ext_vector_type(2)));
typedef unsigned u32x4 __attribute__((ext_vector_type(4)));
typedef unsigned short u16x2 __attribute__((ext_vector_type(2)));
typedef GAS unsigned gu32;
typedef __amdgpu_buffer_rsrc_t rsrc_t;
typedef unsigned u32x8 __attribute__((ext_vector_type(8)));
typedef unsigned u32x16 __attribute__((ext_vector_type(16)));
#define CAS __attribute__((address_space(4)))

constexpr int NR = 8;
constexpr int KD = 2048;
constexpr int NL = 256;
constexpr int ML = 256;
constexpr int NWG = 256;
constexpr int NWAVE = 7;
constexpr int NTHREADS = 64 * NWAVE;
constexpr int NI = 11;
constexpr int NX = 16 - NI;
constexpr int NSEND = 32;
constexpr int CODE_BYTES = 26 * 1024;

constexpr unsigned XBLK = 65536;
constexpr unsigned WGRP = 131072;
constexpr unsigned IN_OFF = 0;
constexpr unsigned CR_OFF = 4 * NI * XBLK;
constexpr unsigned CR_SZ = 2 * WGRP + NX * XBLK;
constexpr unsigned OW_OFF = CR_OFF + 4 * CR_SZ;
constexpr unsigned OX_OFF = OW_OFF + 8 * WGRP;
constexpr unsigned SLOT_BYTES = OX_OFF + (NX + 1) * XBLK;
constexpr size_t WS_CNT  = 0;
constexpr size_t WS_TMO  = 32768;
constexpr size_t WS_TB   = 40960;
constexpr size_t WS_MARK = 65536;
constexpr size_t WS_SLOT = 1u << 20;
constexpr size_t WS_NEED = WS_SLOT + 3 * (size_t)SLOT_BYTES;
constexpr unsigned long long DEADLINE_TICKS = 40000000ull;
constexpr unsigned long long SLOW_TICKS = 20000ull;

constexpr int LDS_BYTES = 96 * 1024;
constexpr int LDS_PART = 1024;
constexpr int LDS_TR = 1024;
constexpr int LDS_TR5 = LDS_PART + 6 * 8192;

constexpr int PP = 115;
constexpr int T1 = 195;
constexpr int T1_IN = 185;
constexpr int X_T0 = T1 + 3 * PP;
constexpr int OUT_T0 = T1 + 8 * PP;
constexpr int CHK_T0 = OUT_T0 + (2 * (64 - NX) * PP >> 5) - 30;
constexpr int XS_LD = 70;
constexpr int TB_LEAD = 10;
constexpr int Z_T0 = 350, Z_P = 170, Z_S = 2, Z_N = 17;
constexpr int WIN_IN = 6, WIN_W = 4, WIN_X = 4, WIN_M = 2;
constexpr int POLL_CHK = 16;
constexpr int POLL_MASK = 3;
constexpr int CHK_BURST = 3;
constexpr int CHK_FAST = 3;
constexpr int CHK_GAP = 3, CHK_SLEEP = 2;
constexpr unsigned OUT_BASE[4] = {80u | 88u << 8 | 96u << 16 | 104u << 24, 48u | 56u << 8 | 64u << 16 | 72u << 24,
                                  24u | 32u << 8 | 40u << 16 | 112u << 24, 0u | 8u << 8 | 16u << 16 | 120u << 24};
constexpr unsigned OUT_EARLY = 12;
constexpr int OUT_SLOTS = 2 * (64 - NX);
struct Order8 { unsigned char e[64]; };
constexpr Order8 make_order() {
    Order8 o{};
    int t[64] = {};
    int n = 0;
    for (int b = 0; b < NI; ++b) { o.e[n] = (unsigned char)b; t[n] = 0; ++n; }
    for (int b = 0; b < NI; ++b) for (int j = 1; j < 4; ++j) { o.e[n] = (unsigned char)(j << 4 | b); t[n] = 10 * (b + 1); ++n; }
    for (int b = NI; b < 16; ++b) for (int c = 0; c < 4; ++c) {
        o.e[n] = (unsigned char)(1 << 6 | c << 4 | b);
        t[n] = 10 * (b - NI) + 50;
        ++n;
    }
    for (int i = 1; i < 64; ++i) {
        const unsigned char e = o.e[i]; const int ti = t[i];
        int j = i - 1;
        while (j >= 0 && t[j] > ti) { o.e[j + 1] = o.e[j]; t[j + 1] = t[j]; --j; }
        o.e[j + 1] = e; t[j + 1] = ti;
    }
    return o;
}

struct Params {
    const float* x;
    const float* w;
    char* ws;
    char* out;
    char* in_ws[3];
    int pa;
    int cls;
    char* cr_ws[4];
    char* cr_out[4];
    unsigned long long blk[76][2];
    unsigned chk[4][32];
};

template <int I, int N, class F> __device__ __forceinline__ void static_for(F&& f) {
    if constexpr (I < N) { f(std::integral_constant<int, I>{}); static_for<I + 1, N>(f); }
}
template <int N> __device__ __forceinline__ void wait_vm() { asm volatile("s_waitcnt vmcnt(%0)" :: "n"(N) : "memory"); }
__device__ __forceinline__ rsrc_t mk_rsrc(const void* p, unsigned bytes) {
    return __builtin_amdgcn_make_buffer_rsrc((void*)p, 0, (int)bytes, 0x00020000);
}
constexpr int AUX_SYS = 17;
__device__ __forceinline__ void st16(u32x4 v, rsrc_t r, unsigned voff, unsigned soff) {
    __builtin_amdgcn_raw_buffer_store_b128(v, r, (int)(voff + soff), 0, AUX_SYS);
}
__device__ __forceinline__ u32x4 ld16(rsrc_t r, unsigned voff, unsigned soff) {
    return __builtin_amdgcn_raw_buffer_load_b128(r, (int)voff, (int)soff, AUX_SYS);
}
__device__ __forceinline__ unsigned ld4(rsrc_t r, unsigned voff, unsigned soff) {
    return __builtin_amdgcn_raw_buffer_load_b32(r, (int)voff, (int)soff, AUX_SYS);
}
__device__ __forceinline__ u32x4 mk_desc(const void* p, unsigned bytes) {
    const unsigned long long a = (unsigned long long)p;
    return (u32x4){(unsigned)a, (unsigned)(a >> 32) & 0xffffu, bytes, 0x00020000u};
}
__device__ __forceinline__ void ld4_into(unsigned& v, u32x4 desc, unsigned voff, unsigned soff) {
    asm volatile("buffer_load_dword %0, %1, %2, %3 offen sc0 sc1" : "+v"(v) : "v"(voff), "s"(desc), "s"(soff) : "memory");
}
template <int N, class T> __device__ __forceinline__ void wait_for(T& v) { asm volatile("s_waitcnt vmcnt(%1)" : "+v"(v) : "n"(N) : "memory"); }
__device__ __forceinline__ bool wave_all(bool c) { return __builtin_amdgcn_ballot_w64(!c) == 0ull; }
__device__ __forceinline__ unsigned pack2(float a, float b) {
    const f16x2 h = __builtin_convertvector((f32x2){a, b}, f16x2);
    const u16x2 u = __builtin_bit_cast(u16x2, h);
    const u16x2 one = {1, 1};
    return (__builtin_bit_cast(unsigned, (u16x2)(u - one)) & 0x80008000u) | __builtin_bit_cast(unsigned, u);
}

#define R_MMA(A0, A1, A2, A3, B0, B3, C0, C3) \
    "v_mfma_f32_16x16x32_f16 v[56:59], v[" #A0 ":" #A3 "], v[" #B0 ":" #B3 "], v[56:59]\n\t" \
    "v_min3_u32 v52, v52, v" #A0 ", v" #A1 "\n\t" \
    "v_mfma_f32_16x16x32_f16 v[60:63], v[" #A0 ":" #A3 "], v[" #C0 ":" #C3 "], v[60:63]\n\t" \
    "v_min3_u32 v53, v53, v" #A2 ", v" #A3 "\n\t"
#define R_MMA0 \
    "v_mfma_f32_16x16x32_f16 v[56:59], v[192:195], v[64:67], 0\n\t" \
    "v_min_u32 v52, v192, v193\n\t" \
    "v_mfma_f32_16x16x32_f16 v[60:63], v[192:195], v[68:71], 0\n\t" \
    "v_min_u32 v53, v194, v195\n\t"
#define R_LD(A0, A3, VO, OFF) "buffer_load_dwordx4 v[" #A0 ":" #A3 "], %[" #VO "], %[desc], %[toff] offen offset:" #OFF " sc0 sc1\n\t"
#define R_W(N) "s_waitcnt vmcnt(" #N ")\n\t"
#define R_BOTH \
    R_W(16) R_MMA0 R_LD(192, 195, vo0, 0) \
    R_W(16) R_MMA(196, 197, 198, 199, 72, 75, 76, 79) R_LD(196, 199, vo0, 1024) \
    R_W(16) R_MMA(200, 201, 202, 203, 80, 83, 84, 87) R_LD(200, 203, vo0, 2048) \
    R_W(16) R_MMA(204, 205, 206, 207, 88, 91, 92, 95) R_LD(204, 207, vo0, 3072) \
    R_W(16) R_MMA(208, 209, 210, 211, 96, 99, 100, 103) R_LD(208, 211, vo1, 0) \
    R_W(16) R_MMA(212, 213, 214, 215, 104, 107, 108, 111) R_LD(212, 215, vo1, 1024) \
    R_W(16) R_MMA(216, 217, 218, 219, 112, 115, 116, 119) R_LD(216, 219, vo1, 2048) \
    R_W(16) R_MMA(220, 221, 222, 223, 120, 123, 124, 127) R_LD(220, 223, vo1, 3072) \
    R_W(16) R_MMA(224, 225, 226, 227, 128, 131, 132, 135) R_LD(224, 227, vo2, 0) \
    R_W(16) R_MMA(228, 229, 230, 231, 136, 139, 140, 143) R_LD(228, 231, vo2, 1024) \
    R_W(16) R_MMA(232, 233, 234, 235, 144, 147, 148, 151) R_LD(232, 235, vo2, 2048) \
    R_W(16) R_MMA(236, 237, 238, 239, 152, 155, 156, 159) R_LD(236, 239, vo2, 3072) \
    R_W(16) R_MMA(240, 241, 242, 243, 160, 163, 164, 167) R_LD(240, 243, vo3, 0) \
    R_W(16) R_MMA(244, 245, 246, 247, 168, 171, 172, 175) R_LD(244, 247, vo3, 1024) \
    R_W(16) R_MMA(248, 249, 250, 251, 176, 179, 180, 183) R_LD(248, 251, vo3, 2048) \
    R_W(16) R_MMA(252, 253, 254, 255, 184, 187, 188, 191) R_LD(252, 255, vo3, 3072)
#define R_MULT \
    R_W(16) R_MMA0 \
    R_W(15) R_MMA(196, 197, 198, 199, 72, 75, 76, 79) \
    R_W(14) R_MMA(200, 201, 202, 203, 80, 83, 84, 87) \
    R_W(13) R_MMA(204, 205, 206, 207, 88, 91, 92, 95) \
    R_W(12) R_MMA(208, 209, 210, 211, 96, 99, 100, 103) \
    R_W(11) R_MMA(212, 213, 214, 215, 104, 107, 108, 111) \
    R_W(10) R_MMA(216, 217, 218, 219, 112, 115, 116, 119) \
    R_W(9) R_MMA(220, 221, 222, 223, 120, 123, 124, 127) \
    R_W(8) R_MMA(224, 225, 226, 227, 128, 131, 132, 135) \
    R_W(7) R_MMA(228, 229, 230, 231, 136, 139, 140, 143) \
    R_W(6) R_MMA(232, 233, 234, 235, 144, 147, 148, 151) \
    R_W(5) R_MMA(236, 237, 238, 239, 152, 155, 156, 159) \
    R_W(4) R_MMA(240, 241, 242, 243, 160, 163, 164, 167) \
    R_W(3) R_MMA(244, 245, 246, 247, 168, 171, 172, 175) \
    R_W(2) R_MMA(248, 249, 250, 251, 176, 179, 180, 183) \
    R_W(1) R_MMA(252, 253, 254, 255, 184, 187, 188, 191)
#define R_FILL \
    R_LD(192, 195, vo0, 0) R_LD(196, 199, vo0, 1024) \
    R_LD(200, 203, vo0, 2048) R_LD(204, 207, vo0, 3072) \
    R_LD(208, 211, vo1, 0) R_LD(212, 215, vo1, 1024) \
    R_LD(216, 219, vo1, 2048) R_LD(220, 223, vo1, 3072) \
    R_LD(224, 227, vo2, 0) R_LD(228, 231, vo2, 1024) \
    R_LD(232, 235, vo2, 2048) R_LD(236, 239, vo2, 3072) \
    R_LD(240, 243, vo3, 0) R_LD(244, 247, vo3, 1024) \
    R_LD(248, 251, vo3, 2048) R_LD(252, 255, vo3, 3072)


__device__ __forceinline__ void gate(unsigned long long t) {
    while ((long long)(__builtin_amdgcn_s_memrealtime() - t) < 0) __builtin_amdgcn_s_sleep(1);
}
__device__ __forceinline__ bool give_up(unsigned long long t0, gu32* tmo, unsigned call, int lane) {
    const unsigned long long dt = __builtin_amdgcn_s_memrealtime() - t0;
    if (dt > SLOW_TICKS) {
        const unsigned tv = __builtin_amdgcn_readfirstlane(__hip_atomic_load(tmo, __ATOMIC_RELAXED, __HIP_MEMORY_SCOPE_AGENT));
        if (tv == call || dt > DEADLINE_TICKS) {
            __hip_atomic_store(tmo, call, __ATOMIC_RELAXED, __HIP_MEMORY_SCOPE_AGENT);
            return true;
        }
    }
    return false;
}

__device__ __forceinline__ unsigned long long time_base(unsigned long long ts, unsigned v) {
    unsigned d = (unsigned)ts - v, m = 0;
    d = d < 100u ? d : 0u;
#pragma unroll
    for (int i = 0; i < 8; ++i) m = max(m, (unsigned)__builtin_amdgcn_readlane((int)d, i));
    return ts - m;
}

__device__ __forceinline__ u32x4 x_fragment(const f32x4 lo, const f32x4 hi, LAS char* tr, int lane) {
    const u32x4 v = {pack2(lo[0], lo[1]), pack2(lo[2], lo[3]), pack2(hi[0], hi[1]), pack2(hi[2], hi[3])};
    *(LAS u32x4*)(tr + (lane >> 2) * 16 + (lane & 3) * 320) = v;
    return *(LAS u32x4*)(tr + (lane & 15) * 16 + (lane >> 4) * 320);
}

__global__ void __launch_bounds__(NTHREADS, 2) ag_gemm(Params P) {
    extern __shared__ __attribute__((aligned(16))) char smem_[];
    LAS char* smem = (LAS char*)smem_;
    const unsigned long long ts = __builtin_amdgcn_s_memrealtime();
    const CAS char* const ka = (const CAS char*)__builtin_amdgcn_kernarg_segment_ptr();
    const u32x16 k0 = *(const volatile CAS u32x16*)ka;
    const u32x8 k1 = *(const volatile CAS u32x8*)(ka + 64);
    static_assert(offsetof(Params, in_ws) == 32 && offsetof(Params, pa) == 56 && offsetof(Params, cr_ws) == 64, "argument layout");
    __builtin_amdgcn_sched_barrier(0);
    const int wg = blockIdx.x;
    const int pwave = __builtin_amdgcn_readfirstlane(threadIdx.x >> 6);
    const int lane = threadIdx.x & 63;
    const bool sender = wg < NSEND;
    const int wave = (int)(((sender ? 0x6543210u : 0x3210645u) >> (4 * pwave)) & 7u);
    const int xcd = wg & 7, u = (wg >> 3) - NSEND / 8, rq = xcd >> 1;
    const bool remote = u < 16;
    const int uu = remote ? u : u - 16;
    const int cg = 4 * (xcd & 1) + (uu & 3), m = uu >> 2;
    {
        static_assert(CODE_BYTES > 3 * 8192 && CODE_BYTES <= 4 * 8192, "four loads of 64 x 128 bytes");
        const unsigned pf = (unsigned)(pwave ^ 6) << 16 | (unsigned)wg;
        unsigned t0, t1, t2, t3;
        asm volatile(
            "s_cmp_ge_u32 %[pf], 8\n\t"
            "s_cbranch_scc1 .Lpf_skip%=\n\t"
            "s_getpc_b64 vcc\n"
            ".Lpf_pc%=:\n\t"
            "s_add_u32 vcc_lo, vcc_lo, _Z7ag_gemm6Params-.Lpf_pc%=\n\t"
            "s_addc_u32 vcc_hi, vcc_hi, -1\n\t"
            "v_lshlrev_b32 %[t0], 7, %[ln]\n\t"
            "v_add_u32 %[t1], 0x2000, %[t0]\n\t"
            "v_add_u32 %[t2], 0x4000, %[t0]\n\t"
            "v_add_u32 %[t3], 0x6000, %[t0]\n\t"
            "v_min_u32 %[t3], %[last], %[t3]\n\t"
            "global_load_dword %[t0], %[t0], vcc\n\t"
            "global_load_dword %[t1], %[t1], vcc\n\t"
            "global_load_dword %[t2], %[t2], vcc\n\t"
            "global_load_dword %[t3], %[t3], vcc\n\t"
            "s_waitcnt vmcnt(0)\n\t"
            "s_endpgm\n"
            ".Lpf_skip%=:"
            : [t0] "=&v"(t0), [t1] "=&v"(t1), [t2] "=&v"(t2), [t3] "=&v"(t3)
            : [pf] "s"(pf), [ln] "v"(lane), [last] "n"(CODE_BYTES - 128) : "scc", "vcc", "memory");
    }
    if (wave == 6 && (sender || remote)) return;
    auto ptr_of = [](unsigned lo, unsigned hi) { return (char*)((unsigned long long)lo | (unsigned long long)hi << 32); };
#define PX ((const float*)ptr_of(k0[0], k0[1]))
#define PW ((const float*)ptr_of(k0[2], k0[3]))
    char* const myws = ptr_of(k0[4], k0[5]);
    char* const myout = ptr_of(k0[6], k0[7]);
    char* const in1 = ptr_of(k0[8], k0[9]); char* const in2 = ptr_of(k0[10], k0[11]); char* const in3 = ptr_of(k0[12], k0[13]);
    const int pa = (int)k0[14], cls = (int)k0[15];
    char* const cr0 = ptr_of(k1[0], k1[1]); char* const cr1 = ptr_of(k1[2], k1[3]);
    char* const cr2 = ptr_of(k1[4], k1[5]); char* const cr3 = ptr_of(k1[6], k1[7]);
    auto cross_ws = [&](int c) { return c == 0 ? cr0 : c == 1 ? cr1 : c == 2 ? cr2 : cr3; };
    const unsigned cnt_off = WS_CNT + wg * 128 + wave * 4;
#define TMO ((gu32*)(myws + WS_TMO))
    unsigned cv;
    gu32* cntp;
    auto load_cv = [&]() {
        cntp = (gu32*)(myws + cnt_off);
        cv = __hip_atomic_load(cntp, __ATOMIC_RELAXED, __HIP_MEMORY_SCOPE_AGENT);
    };
    auto ldx = [&](rsrc_t r, unsigned voff, unsigned soff) { return __builtin_bit_cast(f32x4, __builtin_amdgcn_raw_buffer_load_b128(r, (int)voff, (int)soff, 0)); };
    auto ldw = [&](rsrc_t r, unsigned voff, unsigned soff) { return __builtin_bit_cast(f32x2, __builtin_amdgcn_raw_buffer_load_b64(r, (int)voff, (int)soff, 0)); };
    const unsigned xlane = (lane >> 2) * (KD * 4) + (lane & 3) * 16;
    const unsigned wlane = (lane >> 4) * (4 * NL * 4) + (lane & 15) * 8;
    auto ld_pair = [&](f32x2 (&d)[8], rsrc_t r, unsigned voff, int pr) {
#pragma unroll
        for (int j = 0; j < 8; ++j) d[j] = ldw(r, voff + (j & 3) * (NL * 4) + (pr >> 1) * 128, (j >> 2) * (16 * NL * 4) + (pr & 1) * (32 * 32 * NL * 4));
    };
    const unsigned l16 = lane * 16;
    LAS unsigned* lcnt = (LAS unsigned*)smem;

    if (sender) {
        LAS char* tr = smem + LDS_TR + wave * 1280;
        if (wave < 2) {
            const int ks = wg * 2 + wave;
            unsigned xo = xlane + ks * 128;
            asm volatile("" : "+v"(xo));
            __builtin_amdgcn_sched_barrier(0);
            load_cv();
            const rsrc_t rx = mk_rsrc(PX, ML * KD * 4);
            f32x4 raw[NI][2];
#pragma unroll
            for (int b = 1; b < NI; ++b) {
                raw[b][0] = ldx(rx, xo, b * 16 * KD * 4);
                raw[b][1] = ldx(rx, xo + 64, b * 16 * KD * 4);
            }
            const unsigned tbv = ld4(mk_rsrc(myws + WS_TB, 32), (lane & 7) * 4, 0);
            __builtin_amdgcn_sched_barrier(0);
            __builtin_amdgcn_s_barrier();
            __builtin_amdgcn_sched_barrier(0);
            const unsigned call = __builtin_amdgcn_readfirstlane(cv) + 1u;
            if (lane == 0) __hip_atomic_store(cntp, call, __ATOMIC_RELAXED, __HIP_MEMORY_SCOPE_AGENT);
            const size_t cur = WS_SLOT + (size_t)(call % 3u) * SLOT_BYTES;
            unsigned long long tg = ts + T1_IN + (ks * PP >> 6);
            const rsrc_t r1 = mk_rsrc(in1 + cur, SLOT_BYTES);
            const rsrc_t r2 = mk_rsrc(in2 + cur, SLOT_BYTES);
            const rsrc_t r3 = mk_rsrc(in3 + cur, SLOT_BYTES);
            const unsigned voff = ks * 1024 + l16;
            static_for<1, NI>([&](auto B) {
                constexpr int b = B;
                const u32x4 fr = x_fragment(raw[b][0], raw[b][1], tr, lane);
                if constexpr (b >= 2 && (b - 1) * 3 > WIN_IN) wait_vm<WIN_IN>();
                if constexpr (b == 2) tg -= ts - time_base(ts, tbv);
                gate(tg + (b - 1) * PP);
                const unsigned so = IN_OFF + (pa * NI + b) * XBLK;
                st16(fr, r1, voff, so);
                st16(fr, r2, voff, so);
                st16(fr, r3, voff, so);
                __builtin_amdgcn_sched_barrier(0);
            });
            if (lane == 0) __hip_atomic_fetch_add(&lcnt[0], 1u, __ATOMIC_RELAXED, __HIP_MEMORY_SCOPE_WORKGROUP);
            wait_vm<0>();
            return;
        }
        const int v = wg * 4 + (wave - 2), c = wave - 2;
        constexpr int NXI = (NX * 64 + 127) / 128;
        unsigned xo[NXI];
#pragma unroll
        for (int it = 0; it < NXI; ++it) {
            int item = v + 128 * it;
            if (item >= NX * 64) item = v;
            xo[it] = xlane + (NI + (item >> 6)) * (16 * KD * 4) + (item & 63) * 128;
            asm volatile("" : "+v"(xo[it]));
        }
        __builtin_amdgcn_sched_barrier(0);
        load_cv();
        const rsrc_t rx = mk_rsrc(PX, ML * KD * 4);
        f32x4 xr[NXI][2];
        gate(ts + XS_LD);
#pragma unroll
        for (int it = 0; it < NXI; ++it) {
            xr[it][0] = ldx(rx, xo[it], 0);
            xr[it][1] = ldx(rx, xo[it] + 64, 0);
        }
        const unsigned tbv = ld4(mk_rsrc(myws + WS_TB, 32), (lane & 7) * 4, 0);
        if (wave == 2 && lane == 0) lcnt[0] = 0u;
        asm volatile("s_waitcnt lgkmcnt(0)" ::: "memory");
        __builtin_amdgcn_sched_barrier(0);
        __builtin_amdgcn_s_barrier();
        __builtin_amdgcn_sched_barrier(0);
        __builtin_amdgcn_sched_barrier(0);
        const rsrc_t rmine = mk_rsrc(myout, 2048 * NL * 4);
        const unsigned call = __builtin_amdgcn_readfirstlane(cv) + 1u;
        if (lane == 0) __hip_atomic_store(cntp, call, __ATOMIC_RELAXED, __HIP_MEMORY_SCOPE_AGENT);
        const size_t cur = WS_SLOT + (size_t)(call % 3u) * SLOT_BYTES;
        const unsigned long long tb = time_base(ts, tbv);
        const unsigned long long tx = tb + X_T0 + (v * (2 * PP) >> 7);
        u32x4 xf[NXI];
#pragma unroll
        for (int it = 0; it < NXI; ++it) xf[it] = x_fragment(xr[it][0], xr[it][1], tr, lane);
        __builtin_amdgcn_sched_barrier(0);
        const rsrc_t q0 = mk_rsrc(cr0 + cur, SLOT_BYTES), q1 = mk_rsrc(cr1 + cur, SLOT_BYTES);
        const rsrc_t q2 = mk_rsrc(cr2 + cur, SLOT_BYTES), q3 = mk_rsrc(cr3 + cur, SLOT_BYTES);
        static_for<0, NXI>([&](auto IT) {
            constexpr int it = IT;
            const int item = v + 128 * it;
            if (item < NX * 64) {
                wait_vm<WIN_X>();
                gate(tx + it * (2 * PP));
                const unsigned so = CR_OFF + pa * CR_SZ + 2 * WGRP + (item >> 6) * XBLK + (item & 63) * 1024;
                st16(xf[it], q0, l16, so);
                st16(xf[it], q1, l16, so);
                st16(xf[it], q2, l16, so);
                st16(xf[it], q3, l16, so);
            }
            __builtin_amdgcn_sched_barrier(0);
        });
        wait_vm<0>();
        {
            const unsigned long long t0 = __builtin_amdgcn_s_memrealtime();
            for (unsigned it = 1; __hip_atomic_load(&lcnt[0], __ATOMIC_RELAXED, __HIP_MEMORY_SCOPE_WORKGROUP) < 2u; ++it) {
                __builtin_amdgcn_s_sleep(4);
                if ((it & 255u) == 0u && __builtin_amdgcn_s_memrealtime() - t0 > DEADLINE_TICKS) break;
            }
            bool bad = false;
            const rsrc_t rf = mk_rsrc(myws + WS_MARK, 128 * 128);
            for (unsigned it = 1;; ++it) {
                const unsigned v0 = ld4(rf, lane * 128, 0), v1 = ld4(rf, lane * 128 + 8192, 0);
                asm volatile("" ::: "memory");
                if (wave_all((int)(v0 - call) >= 0 && (int)(v1 - call) >= 0)) break;
                if (give_up(t0, TMO, call, lane)) { bad = true; break; }
            }
            gate(tb + CHK_T0);
            const unsigned chk = P.chk[c][wg & 31];
            unsigned o[4];
#pragma unroll
            for (int k = 0; k < 4; ++k) {
                const unsigned e = (chk >> (8 * k)) & 255u;
                o[k] = ((e & 127u) * 16 + (lane >> 3)) * (NL * 4) + ((2 * c + (e >> 7)) * 32 + (lane & 7) * 4) * 4;
            }
            unsigned pend = (wg & 31) + 96 < OUT_SLOTS ? 15u : 7u;
            bool burst = true;
            for (unsigned it = 1; !bad; ++it) {
                const unsigned sel = burst ? pend : pend & (0u - pend);
                u32x4 a[4][2];
#pragma unroll
                for (int k = 0; k < 4; ++k) {
                    const unsigned off = (sel >> k) & 1u ? o[k] : 0x7f000000u;
                    a[k][0] = ld16(rmine, off, 0);
                    a[k][1] = ld16(rmine, off, 8 * NL * 4);
                }
                asm volatile("" ::: "memory");
                unsigned done = 0u;
#pragma unroll
                for (int k = 0; k < 4; ++k) {
                    const u32x4 s4 = a[k][0] | a[k][1];
                    if (wave_all((int)(s4[0] | s4[1] | s4[2] | s4[3]) >= 0)) done |= 1u << k;
                }
                done &= sel;
                pend &= ~done;
                if (pend == 0u) break;
                if ((pend & (pend - 1u)) == 0u) {
                    const unsigned off = pend == 1u ? o[0] : pend == 2u ? o[1] : pend == 4u ? o[2] : o[3];
                    const u32x4 dm = mk_desc(myout, 2048 * NL * 4);
                    u32x4 pl[CHK_FAST][2];
                    auto issue = [&](u32x4& lo, u32x4& hi) {
                        asm volatile("buffer_load_dwordx4 %0, %2, %3, 0 offen sc0 sc1\n\tbuffer_load_dwordx4 %1, %2, %3, %4 offen sc0 sc1"
                                     : "=&v"(lo), "=&v"(hi) : "v"(off), "s"(dm), "s"(8 * NL * 4) : "memory");
                    };
                    auto look = [&](u32x4& lo, u32x4& hi) {
                        asm volatile("s_waitcnt vmcnt(%2)" : "+v"(lo), "+v"(hi) : "n"(2 * (CHK_FAST - 1)) : "memory");
                        const u32x4 s4 = lo | hi;
                        return wave_all((int)(s4[0] | s4[1] | s4[2] | s4[3]) >= 0);
                    };
                    static_for<0, CHK_FAST>([&](auto K) { issue(pl[K][0], pl[K][1]); if constexpr (K + 1 < CHK_FAST) __builtin_amdgcn_s_sleep(CHK_GAP); });
                    for (unsigned rd = 1;; ++rd) {
                        bool ok = false;
                        static_for<0, CHK_FAST>([&](auto K) {
                            if (!ok) {
                                ok = look(pl[K][0], pl[K][1]);
                                if (!ok) issue(pl[K][0], pl[K][1]);
                            }
                        });
                        if (ok) break;
                        if ((rd & 7u) == 0u && give_up(t0, TMO, call, lane)) { bad = true; break; }
                    }
                    auto land = [&](u32x4& lo, u32x4& hi) { asm volatile("s_waitcnt vmcnt(0)" : "+v"(lo), "+v"(hi) :: "memory"); };
                    static_for<0, CHK_FAST>([&](auto K) { land(pl[K][0], pl[K][1]); });
                    break;
                }
                burst = done != 0u || (it & CHK_BURST) == 0u;
                if ((it & 7u) == 0u && give_up(t0, TMO, call, lane)) bad = true;
                __builtin_amdgcn_s_sleep(CHK_SLEEP);
            }
            if (bad) {
                const float nanv = __builtin_nanf("");
                const u32x4 nv = __builtin_bit_cast(u32x4, (f32x4){nanv, nanv, nanv, nanv});
                st16(nv, rmine, o[0], 0);
            }
        }
        return;
    }

    if (wave == 6) {
        load_cv();
        __builtin_amdgcn_s_barrier();
        __builtin_amdgcn_sched_barrier(0);
        const unsigned call = __builtin_amdgcn_readfirstlane(cv) + 1u;
        if (lane == 0) __hip_atomic_store(cntp, call, __ATOMIC_RELAXED, __HIP_MEMORY_SCOPE_AGENT);
        const rsrc_t rz = mk_rsrc(myws + WS_SLOT + (size_t)((call + 2u) % 3u) * SLOT_BYTES, SLOT_BYTES);
        const u32x4 zero = {0u, 0u, 0u, 0u};
        constexpr int NZW = NWG - NSEND - 128, T = SLOT_BYTES / 1024 / NZW;
        static_assert(T * NZW * 1024 == SLOT_BYTES && T % Z_N == 0, "zeroing shares");
        const int j = uu * 8 + xcd;
        for (int i = 0; i < T; i += Z_N) {
            gate(ts + Z_T0 + (i / Z_N) * Z_P + j * Z_S);
#pragma unroll
            for (int k = 0; k < Z_N; ++k) {
                const unsigned zo = ((i + k) * NZW + j) * 1024;
                __builtin_amdgcn_raw_buffer_store_b128(zero, rz, (int)(l16 + zo), 0, AUX_SYS);
            }
        }
        wait_vm<0>();
        return;
    }
    if (wave == 5) {
        LAS char* tr = smem + LDS_TR5;
        if (lane < 8) lcnt[lane] = 0u;
        if (remote) {
            const int c = m, i = xcd + 8 * (u & 3);
            unsigned wo = wlane + i * (32 * NL * 4) + c * 256;
            asm volatile("" : "+v"(wo));
            __builtin_amdgcn_sched_barrier(0);
            load_cv();
            const rsrc_t rw = mk_rsrc(PW, KD * NL * 4);
            f32x2 wr[4][8];
            ld_pair(wr[0], rw, wo, 0);
            __builtin_amdgcn_sched_barrier(0);
            ld_pair(wr[2], rw, wo, 2);
            ld_pair(wr[1], rw, wo, 1);
            ld_pair(wr[3], rw, wo, 3);
            __builtin_amdgcn_sched_barrier(0);
            asm volatile("s_waitcnt lgkmcnt(0)" ::: "memory");
            __builtin_amdgcn_s_barrier();
            __builtin_amdgcn_sched_barrier(0);
            const unsigned call = __builtin_amdgcn_readfirstlane(cv) + 1u;
            if (lane == 0) __hip_atomic_store(cntp, call, __ATOMIC_RELAXED, __HIP_MEMORY_SCOPE_AGENT);
            const size_t cur = WS_SLOT + (size_t)(call % 3u) * SLOT_BYTES;
            const rsrc_t rl = mk_rsrc(myws + cur, SLOT_BYTES);
            const rsrc_t rc = mk_rsrc(cross_ws(c) + cur, SLOT_BYTES);
            u32x4 f[4][2];
            auto cvt_pair = [&](int pr) {
                f[pr][0] = (u32x4){pack2(wr[pr][0][0], wr[pr][1][0]), pack2(wr[pr][2][0], wr[pr][3][0]), pack2(wr[pr][4][0], wr[pr][5][0]), pack2(wr[pr][6][0], wr[pr][7][0])};
                f[pr][1] = (u32x4){pack2(wr[pr][0][1], wr[pr][1][1]), pack2(wr[pr][2][1], wr[pr][3][1]), pack2(wr[pr][4][1], wr[pr][5][1]), pack2(wr[pr][6][1], wr[pr][7][1])};
            };
            auto send_pair = [&](int pr) {
                const unsigned so = CR_OFF + pa * CR_SZ + (pr >> 1) * WGRP + (i + 32 * (pr & 1)) * 2048;
                st16(f[pr][0], rc, l16, so);
                st16(f[pr][1], rc, l16, so + 1024);
            };
            cvt_pair(0);
            __builtin_amdgcn_sched_barrier(0);
            const unsigned tbv = ld4(mk_rsrc(myws + WS_TB, 32), (lane & 7) * 4, 0);
            __builtin_amdgcn_sched_barrier(0);
            send_pair(0);
            __builtin_amdgcn_sched_barrier(0);
            cvt_pair(2); cvt_pair(1); cvt_pair(3);
            __builtin_amdgcn_sched_barrier(0);
            const rsrc_t rmine = mk_rsrc(myout, 2048 * NL * 4);
            auto mark_tile = [&](int g, int t) {
                const int oc = cls ^ 4;
                const int rb = t < 4 * NI ? (oc | (t / NI)) * 16 + t % NI : (cls | ((t - 4 * NI) / NX)) * 16 + NI + (t - 4 * NI) % NX;
                const bool mine = t >= 4 * NI && (t - 4 * NI) / NX == pa;
                const unsigned o = mine ? 0x7f000000u : (unsigned)(rb * 16 + (lane >> 3)) * (NL * 4) + (g * 32 + (lane & 7) * 4) * 4;
                const u32x4 mk = {0xffffffffu, 0xffffffffu, 0xffffffffu, 0xffffffffu};
                st16(mk, rmine, o, 0);
                st16(mk, rmine, o, 8 * NL * 4);
            };
            mark_tile(2 * c, i);
            mark_tile(2 * c, i + 32);
            mark_tile(2 * c + 1, i);
            mark_tile(2 * c + 1, i + 32);
#pragma unroll
            for (int pr = 0; pr < 4; ++pr) {
                const unsigned so = OW_OFF + (2 * c + (pr >> 1)) * WGRP + (i + 32 * (pr & 1)) * 2048;
                st16(f[pr][0], rl, l16, so);
                st16(f[pr][1], rl, l16, so + 1024);
            }
            __builtin_amdgcn_sched_barrier(0);
            const unsigned long long tw = time_base(ts, tbv) + T1 + (i * PP >> 5);
            gate(tw); send_pair(2);
            __builtin_amdgcn_sched_barrier(0);
            wait_vm<WIN_M>();
            if (lane == 0) __hip_atomic_store((gu32*)(myws + WS_MARK + (size_t)(i * 4 + c) * 128), call, __ATOMIC_RELAXED, __HIP_MEMORY_SCOPE_SYSTEM);
            gate(tw + PP); send_pair(1);
            __builtin_amdgcn_sched_barrier(0);
            wait_vm<WIN_W>(); gate(tw + 2 * PP); send_pair(3);
            __builtin_amdgcn_sched_barrier(0);
            if (lane == 0) __hip_atomic_store(&lcnt[7], 1u, __ATOMIC_RELAXED, __HIP_MEMORY_SCOPE_WORKGROUP);
            return;
        } else {
            static_assert(NI == 11, "blocks of the twelve own-column workgroups of an XCD");
            const bool has0 = uu < 8, has2 = uu >= 8 && uu < 11;
            const int b = uu == 0 ? NI : uu == 11 ? NI + 1 : uu, b2 = uu + 5;
            const int ks0 = 2 * xcd + (uu & 1) + ((uu & 7) >> 1) * 16;
            const unsigned kb = b < NI ? 2 * xcd : 4 * xcd, k1 = b < NI ? 16 : 2, k2 = 32;
            unsigned xo0 = xlane + ks0 * 128, xo = xlane + b * (16 * KD * 4) + kb * 128;
            unsigned xo2 = xlane + (has2 ? b2 : b) * (16 * KD * 4) + 4 * xcd * 128;
            unsigned xmid = xo + k1 * 128;
            asm volatile("" : "+v"(xo0), "+v"(xo), "+v"(xo2), "+v"(xmid));
            __builtin_amdgcn_sched_barrier(0);
            if (uu == 11 && lane == 0) __hip_atomic_store((gu32*)(myws + WS_TB + xcd * 4), (unsigned)ts - TB_LEAD, __ATOMIC_RELAXED, __HIP_MEMORY_SCOPE_AGENT);
            __builtin_amdgcn_sched_barrier(0);
            load_cv();
            const rsrc_t rx = mk_rsrc(PX, ML * KD * 4);
            const f32x4 z0 = ldx(rx, xo0, 0), z1 = ldx(rx, xo0 + 64, 0);
            f32x4 xr[8][2], xs[8][2];
#pragma unroll
            for (int it = 0; it < 8; ++it) {
                const unsigned base = (it & 2) ? xmid : xo;
                xr[it][0] = ldx(rx, base + (it & 1) * 128, (it >> 2) * (k2 * 128));
                xr[it][1] = ldx(rx, base + (it & 1) * 128 + 64, (it >> 2) * (k2 * 128));
            }
            if (has2) {
#pragma unroll
                for (int it = 0; it < 8; ++it) {
                    xs[it][0] = ldx(rx, xo2 + (it & 3) * 128, (it >> 2) * (32 * 128));
                    xs[it][1] = ldx(rx, xo2 + (it & 3) * 128 + 64, (it >> 2) * (32 * 128));
                }
            }
            __builtin_amdgcn_sched_barrier(0);
            asm volatile("s_waitcnt lgkmcnt(0)" ::: "memory");
            __builtin_amdgcn_s_barrier();
            __builtin_amdgcn_sched_barrier(0);
            const unsigned call = __builtin_amdgcn_readfirstlane(cv) + 1u;
            if (lane == 0) __hip_atomic_store(cntp, call, __ATOMIC_RELAXED, __HIP_MEMORY_SCOPE_AGENT);
            const size_t cur = WS_SLOT + (size_t)(call % 3u) * SLOT_BYTES;
            const rsrc_t rl = mk_rsrc(myws + cur, SLOT_BYTES);
            if (has0) {
                const rsrc_t r1 = mk_rsrc(in1 + cur, SLOT_BYTES);
                const rsrc_t r2 = mk_rsrc(in2 + cur, SLOT_BYTES);
                const rsrc_t r3 = mk_rsrc(in3 + cur, SLOT_BYTES);
                const u32x4 fr = x_fragment(z0, z1, tr, lane);
                const unsigned so = IN_OFF + (pa * NI) * XBLK + ks0 * 1024;
                st16(fr, r1, l16, so);
                st16(fr, r2, l16, so);
                st16(fr, r3, l16, so);
                st16(fr, rl, l16, so);
            }
            __builtin_amdgcn_sched_barrier(0);
            {
                const unsigned sb = (b < NI ? IN_OFF + (pa * NI + b) * XBLK : OX_OFF + (b - NI) * XBLK) + kb * 1024;
#pragma unroll
                for (int it = 0; it < 8; ++it) {
                    const u32x4 fr = x_fragment(xr[it][0], xr[it][1], tr, lane);
                    st16(fr, rl, l16, sb + ((it & 1) + ((it >> 1) & 1) * k1 + (it >> 2) * k2) * 1024);
                }
            }
            if (has2) {
                const unsigned sb = OX_OFF + (b2 - NI) * XBLK + 4 * xcd * 1024;
#pragma unroll
                for (int it = 0; it < 8; ++it) {
                    const u32x4 fr = x_fragment(xs[it][0], xs[it][1], tr, lane);
                    st16(fr, rl, l16, sb + ((it & 3) + (it >> 2) * 32) * 1024);
                }
            }
        }
        __builtin_amdgcn_sched_barrier(0);
        if (lane == 0) __hip_atomic_store(&lcnt[7], 1u, __ATOMIC_RELAXED, __HIP_MEMORY_SCOPE_WORKGROUP);
        return;
    }
    static_assert(CR_OFF % XBLK == 0 && CR_SZ % XBLK == 0 && WGRP % XBLK == 0 && SLOT_BYTES / XBLK <= 255, "block places");
    const int cfg = (remote ? (cg >> 1) * 16 : 64) + m * 4 + rq;
    const unsigned long long dplace = P.blk[cfg][0], drow = P.blk[cfg][1];
    const int nblk = (int)(drow >> 56);
    load_cv();
    auto block_of = [&](int t, unsigned& xoff, unsigned& row0) {
        xoff = ((unsigned)(dplace >> (8 * t)) & 255u) * XBLK;
        row0 = ((unsigned)(drow >> (8 * t)) & 255u) * 16u;
    };

    if (wave == 4) {
        __builtin_amdgcn_sched_barrier(0);
        __builtin_amdgcn_s_barrier();
        __builtin_amdgcn_sched_barrier(0);
        const unsigned call = __builtin_amdgcn_readfirstlane(cv) + 1u;
        if (lane == 0) __hip_atomic_store(cntp, call, __ATOMIC_RELAXED, __HIP_MEMORY_SCOPE_AGENT);
        unsigned long long tb = ts;
        if (remote) {
            for (unsigned it = 1; __hip_atomic_load(&lcnt[7], __ATOMIC_RELAXED, __HIP_MEMORY_SCOPE_WORKGROUP) == 0u; ++it) {
                __builtin_amdgcn_s_sleep(2);
                if ((it & 255u) == 0u && __builtin_amdgcn_s_memrealtime() - ts > DEADLINE_TICKS) break;
            }
            tb = time_base(ts, ld4(mk_rsrc(myws + WS_TB, 32), (lane & 7) * 4, 0));
        }
        const int h = cg & 1;
        const unsigned col0 = remote ? 64 * pa + 32 * h : 32 * cg;
        const rsrc_t ro = mk_rsrc(remote ? P.cr_out[(cg >> 1) & 3] : myout, 2048 * NL * 4);
        const unsigned ovoff = (lane >> 3) * (NL * 4) + (col0 + (lane & 7) * 4) * 4;
        const unsigned obase = (unsigned)(dplace >> 32);
        const bool defer = remote && !((OUT_EARLY >> m) & 1u);
        auto reduce = [&](int t, f32x4& r0, f32x4& r1) {
            bool bad = false;
            if (__hip_atomic_load(&lcnt[t], __ATOMIC_ACQUIRE, __HIP_MEMORY_SCOPE_WORKGROUP) < 4u) {
                const unsigned long long t0 = __builtin_amdgcn_s_memrealtime();
                for (unsigned it = 1; __hip_atomic_load(&lcnt[t], __ATOMIC_ACQUIRE, __HIP_MEMORY_SCOPE_WORKGROUP) < 4u; ++it)
                    if ((it & 255u) == 0u && __builtin_amdgcn_s_memrealtime() - t0 > 4 * DEADLINE_TICKS) { bad = true; break; }
            }
            LAS char* part = smem + LDS_PART + t * 8192 + l16;
            f32x4 v[8];
#pragma unroll
            for (int i = 0; i < 8; ++i) v[i] = *(LAS f32x4*)(part + (i >> 2) * 1024 + (i & 3) * 2048);
            r0 = (v[0] + v[1]) + (v[2] + v[3]);
            r1 = (v[4] + v[5]) + (v[6] + v[7]);
#pragma unroll
            for (int r = 0; r < 4; ++r) {
                r0[r] = __builtin_fabsf(r0[r] < 0.f ? 0.f : r0[r]);
                r1[r] = __builtin_fabsf(r1[r] < 0.f ? 0.f : r1[r]);
                if (bad) { r0[r] = __builtin_nanf(""); r1[r] = r0[r]; }
            }
        };
        auto send = [&](int t, const f32x4& r0, const f32x4& r1) {
            unsigned xoff, row0;
            block_of(t, xoff, row0);
            const unsigned o = ovoff + row0 * (NL * 4);
            if (remote) {
                gate(tb + OUT_T0 + ((((obase >> (8 * t)) & 255u) * 2 + h) * PP >> 5));
                st16(__builtin_bit_cast(u32x4, r0), ro, o, 0);
                st16(__builtin_bit_cast(u32x4, r1), ro, o, 8 * NL * 4);
            } else {
                __builtin_amdgcn_raw_buffer_store_b128(__builtin_bit_cast(u32x4, r0), ro, (int)o, 0, 0);
                __builtin_amdgcn_raw_buffer_store_b128(__builtin_bit_cast(u32x4, r1), ro, (int)(o + 8 * NL * 4), 0, 0);
            }
        };
        if (defer) {
            f32x4 res[4][2];
            static_for<0, 4>([&](auto T) { if (T < nblk) reduce(T, res[T][0], res[T][1]); });
            static_for<0, 4>([&](auto T) { if (T < nblk) send(T, res[T][0], res[T][1]); });
        } else {
            for (int t = 0; t < nblk; ++t) {
                f32x4 r0, r1;
                reduce(t, r0, r1);
                send(t, r0, r1);
            }
        }
        if (remote) wait_vm<0>();
        return;
    }

    __builtin_amdgcn_s_barrier();
    __builtin_amdgcn_sched_barrier(0);
    const unsigned call = __builtin_amdgcn_readfirstlane(cv) + 1u;
    if (lane == 0) __hip_atomic_store(cntp, call, __ATOMIC_RELAXED, __HIP_MEMORY_SCOPE_AGENT);
    const size_t cur = WS_SLOT + (size_t)(call % 3u) * SLOT_BYTES;
    const int q = wave;
    const unsigned woff = remote ? CR_OFF + (cg >> 1) * CR_SZ + (cg & 1) * WGRP : OW_OFF + cg * WGRP;
    const rsrc_t rs = mk_rsrc(myws + cur, SLOT_BYTES);
    const u32x4 dfull = mk_desc(myws + cur, SLOT_BYTES);
    const unsigned vo0 = l16 + q * 16384, vo1 = vo0 + 4096, vo2 = vo0 + 8192, vo3 = vo0 + 12288;
    const unsigned xpoll = (q * 16 + (lane & POLL_MASK)) * 1024 + 1020;
    bool bad = false;
    f16x8 b0[16], b1[16];
    u32x4 av[16];
    unsigned pv;
    unsigned long long t0 = __builtin_amdgcn_s_memrealtime();
    for (unsigned it = 1; __hip_atomic_load(&lcnt[7], __ATOMIC_RELAXED, __HIP_MEMORY_SCOPE_WORKGROUP) == 0u; ++it) {
        __builtin_amdgcn_s_sleep(1);
        if ((it & 255u) == 0u && __builtin_amdgcn_s_memrealtime() - t0 > DEADLINE_TICKS) break;
    }
#pragma unroll
    for (int ks = 0; ks < 16; ++ks) av[ks] = (u32x4){0u, 0u, 0u, 0u};
    {
        unsigned xoff, row0;
        const unsigned wpoll = (q * 32 + (lane & 31)) * 1024 + 1020;
        for (unsigned it = 1;; ++it) {
            const unsigned v = ld4(rs, wpoll, woff);
            asm volatile("" ::: "memory");
            if (bad || wave_all(v != 0u)) break;
            if ((it & (POLL_CHK - 1)) == 0u && give_up(t0, TMO, call, lane)) { bad = true; break; }
            __builtin_amdgcn_s_sleep(2);
        }
        block_of(0, xoff, row0);
        for (;;) {
            pv = ld4(rs, xpoll, xoff);
#pragma unroll
            for (int ks = 0; ks < 16; ++ks) {
                const u32x4 w0 = ld16(rs, l16, woff + (q * 32 + ks * 2) * 1024);
                const u32x4 w1 = ld16(rs, l16, woff + (q * 32 + ks * 2 + 1) * 1024);
                b0[ks] = __builtin_bit_cast(f16x8, w0);
                b1[ks] = __builtin_bit_cast(f16x8, w1);
            }
            unsigned ma = 0xffffffffu, mb = 0xffffffffu;
#pragma unroll
            for (int ks = 0; ks < 16; ++ks) {
                const u32x4 w0 = __builtin_bit_cast(u32x4, b0[ks]), w1 = __builtin_bit_cast(u32x4, b1[ks]);
                ma = min(min(ma, w0[0]), w0[1]); ma = min(min(ma, w0[2]), w0[3]);
                mb = min(min(mb, w1[0]), w1[1]); mb = min(min(mb, w1[2]), w1[3]);
            }
            asm volatile("" ::: "memory");
            if (bad || wave_all(ma != 0u && mb != 0u)) break;
            if (give_up(t0, TMO, call, lane)) { bad = true; break; }
        }
    }
    int cur_real = 0;
    int t = 0;
    t0 = __builtin_amdgcn_s_memrealtime();
    __builtin_amdgcn_s_waitcnt(0x0F70);
    for (;;) {
        const int tn = t + cur_real;
        int ld_real = 0;
        unsigned toff = 0, trow;
        if (cur_real) wait_vm<16>(); else wait_vm<0>();
        asm volatile("" : "+v"(pv));
        if (tn < nblk) {
            block_of(tn, toff, trow);
            toff = __builtin_amdgcn_readfirstlane(toff);
            if (!cur_real) {
                for (unsigned it = 1; !bad && !wave_all(pv != 0u); ++it) {
                    if ((it & (POLL_CHK - 1)) == 0u && give_up(t0, TMO, call, lane)) { bad = true; break; }
                    ld4_into(pv, dfull, xpoll, toff);
                    wait_for<0>(pv);
                }
                ld_real = 1;
            } else {
                ld_real = wave_all(pv != 0u);
            }
        }
        ld_real = __builtin_amdgcn_readfirstlane(ld_real);
        {
            const int tp = tn + ld_real;
            unsigned poff, prow;
            block_of(tp < nblk ? tp : nblk - 1, poff, prow);
            ld4_into(pv, dfull, xpoll, __builtin_amdgcn_readfirstlane(poff));
        }
        f32x4 acc0, acc1;
        __builtin_amdgcn_sched_barrier(0);
        unsigned mn0, mn1;
        const int mode = cur_real ? (ld_real ? 1 : 2) : 0;
        asm volatile(
            "s_cmp_lg_u32 %[mode], 1\n\t"
            "s_cbranch_scc1 .Lr_not_both%=\n\t"
            R_BOTH
            "s_nop 15\n\ts_nop 3\n\t"
            "s_branch .Lr_done%=\n"
            ".Lr_not_both%=:\n\t"
            "s_cmp_eq_u32 %[mode], 0\n\t"
            "s_cbranch_scc1 .Lr_fill%=\n\t"
            R_MULT
            "s_nop 15\n\ts_nop 3\n\t"
            "s_branch .Lr_done%=\n"
            ".Lr_fill%=:\n\t"
            R_FILL
            "\n.Lr_done%=:"
            : "+{v[192:195]}"(av[0]), "+{v[196:199]}"(av[1]), "+{v[200:203]}"(av[2]), "+{v[204:207]}"(av[3]), "+{v[208:211]}"(av[4]),
              "+{v[212:215]}"(av[5]), "+{v[216:219]}"(av[6]), "+{v[220:223]}"(av[7]), "+{v[224:227]}"(av[8]), "+{v[228:231]}"(av[9]),
              "+{v[232:235]}"(av[10]), "+{v[236:239]}"(av[11]), "+{v[240:243]}"(av[12]), "+{v[244:247]}"(av[13]), "+{v[248:251]}"(av[14]),
              "+{v[252:255]}"(av[15]),
              "=&{v[56:59]}"(acc0), "=&{v[60:63]}"(acc1), "=&{v52}"(mn0), "=&{v53}"(mn1)
            : "{v[64:67]}"(b0[0]), "{v[68:71]}"(b1[0]), "{v[72:75]}"(b0[1]), "{v[76:79]}"(b1[1]), "{v[80:83]}"(b0[2]), "{v[84:87]}"(b1[2]),
              "{v[88:91]}"(b0[3]), "{v[92:95]}"(b1[3]), "{v[96:99]}"(b0[4]), "{v[100:103]}"(b1[4]), "{v[104:107]}"(b0[5]),
              "{v[108:111]}"(b1[5]), "{v[112:115]}"(b0[6]), "{v[116:119]}"(b1[6]), "{v[120:123]}"(b0[7]), "{v[124:127]}"(b1[7]),
              "{v[128:131]}"(b0[8]), "{v[132:135]}"(b1[8]), "{v[136:139]}"(b0[9]), "{v[140:143]}"(b1[9]), "{v[144:147]}"(b0[10]),
              "{v[148:151]}"(b1[10]), "{v[152:155]}"(b0[11]), "{v[156:159]}"(b1[11]), "{v[160:163]}"(b0[12]), "{v[164:167]}"(b1[12]),
              "{v[168:171]}"(b0[13]), "{v[172:175]}"(b1[13]), "{v[176:179]}"(b0[14]), "{v[180:183]}"(b1[14]), "{v[184:187]}"(b0[15]),
              "{v[188:191]}"(b1[15]),
              [vo0] "v"(vo0), [vo1] "v"(vo1), [vo2] "v"(vo2), [vo3] "v"(vo3), [desc] "s"(dfull), [toff] "s"(toff), [mode] "s"(mode)
            : "memory", "scc");
        if (!cur_real) {
            cur_real = 1;
            continue;
        }
        const bool whole = wave_all(mn0 != 0u && mn1 != 0u);
        if (!bad && !whole) {
            if (give_up(t0, TMO, call, lane)) bad = true;
            else {
                cur_real = 0;
                wait_for<0>(pv);
                pv = 1u;
                {
                    unsigned poff, prow;
                    block_of(t, poff, prow);
                    poff = __builtin_amdgcn_readfirstlane(poff);
                    const unsigned xall = (q * 16 + (lane & 15)) * 1024 + 1020;
                    for (unsigned it = 1;; ++it) {
                        ld4_into(pv, dfull, xall, poff);
                        wait_for<0>(pv);
                        if (wave_all(pv != 0u)) break;
                        if ((it & (POLL_CHK - 1)) == 0u && give_up(t0, TMO, call, lane)) { pv = 1u; break; }
                    }
                }
                continue;
            }
        }
        if (bad) {
            const float nanv = __builtin_nanf("");
            acc0 = (f32x4){nanv, nanv, nanv, nanv};
            acc1 = acc0;
        }
        LAS char* part = smem + LDS_PART + t * 8192 + q * 2048 + (lane >> 4) * 512 + (lane & 15) * 8;
#pragma unroll
        for (int r = 0; r < 4; ++r) *(LAS f32x2*)(part + r * 128) = (f32x2){acc0[r], acc1[r]};
        asm volatile("s_waitcnt lgkmcnt(0)" ::: "memory");
        if (lane == 0) __hip_atomic_fetch_add(&lcnt[t], 1u, __ATOMIC_RELEASE, __HIP_MEMORY_SCOPE_WORKGROUP);
        ++t;
        if (t == nblk) break;
        cur_real = ld_real;
        t0 = __builtin_amdgcn_s_memrealtime();
    }
    wait_vm<0>();
}

extern "C" void kernel_launch(int rank, int nranks, ncclComm_t comm,
                              hipStream_t stream,
                              void* const* d_in, const int* in_sizes, int n_in,
                              void* d_out, int out_size,
                              const int* in_dtypes, int out_dtype,
                              void* const* d_out_all,
                              void* const* d_ws_all, size_t ws_bytes) {
    (void)comm; (void)in_dtypes; (void)out_dtype; (void)d_out;
    if (nranks != NR || n_in != 2 || in_sizes[0] != ML * KD || in_sizes[1] != KD * NL ||
        out_size != NR * ML * NL || ws_bytes < WS_NEED || (unsigned)rank >= (unsigned)NR) {
        fprintf(stderr, "kernel_launch: unexpected shapes\n");
        return;
    }
    static bool attr_set[64] = {};
    int dev = 0;
    HIP_CHECK(hipGetDevice(&dev));
    if (dev >= 0 && dev < 64 && !attr_set[dev]) {
        HIP_CHECK(hipFuncSetAttribute((const void*)ag_gemm, hipFuncAttributeMaxDynamicSharedMemorySize, LDS_BYTES));
        attr_set[dev] = true;
    }
    const int pa = rank & 3, cls = rank & 4, oc = cls ^ 4;
    Params p;
    p.x = (const float*)d_in[0];
    p.w = (const float*)d_in[1];
    p.ws = (char*)d_ws_all[rank];
    p.out = (char*)d_out_all[rank];
    for (int j = 0; j < 3; ++j) p.in_ws[j] = (char*)d_ws_all[cls | ((pa + 1 + j) & 3)];
    for (int c = 0; c < 4; ++c) { p.cr_ws[c] = (char*)d_ws_all[oc | c]; p.cr_out[c] = (char*)d_out_all[oc | c]; }
    p.pa = pa;
    p.cls = cls;
    constexpr Order8 order = make_order();
    auto place_of = [&](unsigned e) {
        const unsigned src = (e >> 4) & 3u, b = e & 15u, a = (pa + src) & 3u;
        return (e >> 6) ? (CR_OFF + 2 * WGRP) / XBLK + src * (CR_SZ / XBLK) + (b - NI) : IN_OFF / XBLK + a * NI + b;
    };
    auto row_of = [&](unsigned e, int mycls, int mypa) {
        const unsigned src = (e >> 4) & 3u, b = e & 15u;
        return (e >> 6) ? ((mycls ^ 4) | src) * (ML / 16) + b : (mycls | ((mypa + src) & 3u)) * (ML / 16) + b;
    };
    auto out_list = [&](int c, int (&pos)[4][4][4]) {
        int n = 0;
        for (int base = 0; base < 128; base += 8)
            for (int m = 0; m < 4; ++m) for (int t = 0; t < 4; ++t) if ((int)((OUT_BASE[m] >> (8 * t)) & 255u) == base)
                for (int rq = 0; rq < 4; ++rq) {
                    const unsigned e = order.e[(t < 3 ? 3 - m + 4 * t : m + 12) * 4 + rq];
                    pos[m][t][rq] = ((e >> 6) && (int)((e >> 4) & 3u) == c) ? -1 : n++;
                }
        return n;
    };
    for (int c = 0; c < 4; ++c) {
        int pos[4][4][4];
        out_list(c, pos);
        for (int m = 0; m < 4; ++m) for (int rq = 0; rq < 4; ++rq) {
            unsigned long long dplace = 0, drow = 0;
            int n = 0;
            for (int t = 0; t < 4; ++t) {
                if (pos[m][t][rq] < 0) continue;
                const unsigned e = order.e[(t < 3 ? 3 - m + 4 * t : m + 12) * 4 + rq];
                dplace |= (unsigned long long)place_of(e) << (8 * n) | (unsigned long long)pos[m][t][rq] << (32 + 8 * n);
                drow |= (unsigned long long)row_of(e, cls, pa) << (8 * n);
                ++n;
            }
            p.blk[c * 16 + m * 4 + rq][0] = dplace;
            p.blk[c * 16 + m * 4 + rq][1] = drow | (unsigned long long)n << 56;
        }
    }
    {
        unsigned place[72], row[72];
        int n = 0;
        for (int b = 0; b < 16; ++b) { place[n] = b < NI ? IN_OFF / XBLK + pa * NI + b : OX_OFF / XBLK + (b - NI); row[n] = rank * (ML / 16) + b; ++n; }
        for (int i = NI; i < 64; ++i) { place[n] = place_of(order.e[i]); row[n] = row_of(order.e[i], cls, pa); ++n; }
        for (int m = 0; m < 3; ++m) for (int rq = 0; rq < 4; ++rq) {
            unsigned long long dplace = 0, drow = 0;
            int k = 0;
            for (int i = m * 4 + rq; i < n; i += 12) { dplace |= (unsigned long long)place[i] << (8 * k); drow |= (unsigned long long)row[i] << (8 * k); ++k; }
            p.blk[64 + m * 4 + rq][0] = dplace;
            p.blk[64 + m * 4 + rq][1] = drow | (unsigned long long)k << 56;
        }
    }
    for (int c = 0; c < 4; ++c) {
        int pos[4][4][4];
        out_list(pa, pos);
        for (int j = 0; j < 32; ++j) p.chk[c][j] = 0u;
        for (int m = 0; m < 4; ++m) for (int t = 0; t < 4; ++t) for (int rq = 0; rq < 4; ++rq) for (int h = 0; h < 2; ++h) {
            if (pos[m][t][rq] < 0) continue;
            const int slot = pos[m][t][rq] * 2 + h;
            const unsigned e = order.e[(t < 3 ? 3 - m + 4 * t : m + 12) * 4 + rq];
            p.chk[c][slot & 31] |= (row_of(e, oc, c) | (unsigned)h << 7) << (8 * (slot >> 5));
        }
    }
    hipLaunchKernelGGL(ag_gemm, dim3(NWG), dim3(NTHREADS), LDS_BYTES, stream, p);
    HIP_CHECK(hipGetLastError());
}
```
